```python
import numpy as np

kernel_name = "gpu_ag_gemm_m4096_k4096_n4096_e4m3e5m2_dqrelu"


ABSMAX_RTOL = 0.05

COMPUTE_DTYPE = "float32"
EPILOGUE = "dequant_relu"
IN_QUANT = None
OUT_QUANT = None
IN_FP8 = "e4m3e5m2"
IN_DTYPES = {"x": "e4m3", "w_mat": "e5m2",
             "scale_x": "float32", "scale_w": "float32"}
WORLD_SIZE = 8

def _snap_fp8(x, mant_bits, max_abs, min_normal):
    a = np.abs(x.astype(np.float64))
    _m, _e = np.frexp(a)
    q = np.where(a >= min_normal, np.ldexp(1.0, _e - 1 - mant_bits),
                 min_normal * 2.0 ** -mant_bits)
    snapped = np.minimum(np.round(a / q) * q, max_abs)
    return (np.sign(x) * snapped).astype(np.float32)

def setup_inputs():

    x = _snap_fp8(np.random.randn(4096, 4096).astype(np.float32), 3, 448.0, 2.0 ** -6)
    w_mat = _snap_fp8(np.random.randn(4096, 4096).astype(np.float32), 2, 57344.0, 2.0 ** -14)
    scale_x = (0.02 * (1.0 + np.random.rand(1))).astype(np.float32)
    scale_w = (0.02 * (1.0 + np.random.rand(1))).astype(np.float32)
    return {"x": x, "w_mat": w_mat, "scale_x": scale_x, "scale_w": scale_w}

def reference(x, w_mat, scale_x, scale_w):
    acc = (x @ w_mat).astype(np.float32)
    y = (acc * (scale_x * scale_w)).astype(np.float32)
    return np.maximum(y, 0.0).astype(np.float32)
```

```cpp
#include <hip/hip_runtime.h>
#include <rccl/rccl.h>
#include <cstdio>
#include <cstdint>

#define HIP_CHECK(expr) do { hipError_t _e = (expr); if (_e != hipSuccess) { \
    fprintf(stderr, "HIP error %s at %s:%d\n", hipGetErrorString(_e), __FILE__, __LINE__); \
    return; } } while (0)

namespace {

typedef int v2i __attribute__((ext_vector_type(2)));
typedef int v4i __attribute__((ext_vector_type(4)));
typedef int v8i __attribute__((ext_vector_type(8)));
typedef float v4f __attribute__((ext_vector_type(4)));
typedef unsigned u32x4 __attribute__((ext_vector_type(4)));
#define LAS __attribute__((address_space(3)))

constexpr int NR = 8;
constexpr int MS = 512;
constexpr int KD = 4096;
constexpr int NS = 512;
constexpr int BM = 64, BN = 128, BK = 128, NSTEP = KD / BK;
constexpr int A_BYTES = BM * BK, B_BYTES = BK * BN, STAGE = A_BYTES + B_BYTES;
constexpr int PD = 3;
constexpr int NSLOT = PD + 1;
constexpr int LDS_BYTES = NSLOT * STAGE + 64;
constexpr int NWG = 256;
constexpr size_t SHARD_BYTES = (size_t)MS * KD;
constexpr size_t EXP_OFF = 0;
constexpr size_t FLAG_OFF = 2 * SHARD_BYTES;
constexpr size_t SEQ_OFF = FLAG_OFF + 65536;
constexpr size_t TMO_OFF = FLAG_OFF + 131072;
constexpr size_t WS_NEED = TMO_OFF + 256;
constexpr unsigned long long DEADLINE_TICKS = 100000000ull;

struct Params {
  const unsigned char* x;
  const unsigned char* w;
  const float* sx;
  const float* sw;
  float* out;
  unsigned char* ws[NR];
  int rank;
};

__device__ __forceinline__ void glds16(const void* g, unsigned lds_addr) {
  unsigned keep;
  asm volatile("s_mov_b32 %0, m0\n\ts_mov_b32 m0, %2\n\ts_nop 0\n\tglobal_load_lds_dwordx4 %1, off\n\ts_mov_b32 m0, %0"
               : "=&s"(keep) : "v"(g), "s"(lds_addr) : "memory");
}
__device__ __forceinline__ void st16x2_sys(void* p, u32x4 v0, u32x4 v1) {
  asm volatile("global_store_dwordx4 %0, %2, off sc0 sc1\n\tglobal_store_dwordx4 %1, %3, off sc0 sc1\n\ts_nop 1"
               :: "v"(p), "v"((unsigned char*)p + 4096), "v"(v0), "v"(v1) : "memory");
}
#define WAIT_VM(n) asm volatile("s_waitcnt vmcnt(" #n ")" ::: "memory")

__global__ __launch_bounds__(256, 1) void ag_gemm_kernel(Params p) {
  extern __shared__ __attribute__((aligned(16))) unsigned char smem[];
  const int tid = threadIdx.x;
  const int lane = tid & 63;
  const int w = __builtin_amdgcn_readfirstlane(tid >> 6);
  const int b = blockIdx.x;
  const int grp = b & 7, idx = b >> 3, rb = idx >> 2, cb = idx & 3;
  const int rank = p.rank;
  const unsigned src = (unsigned)(rank + grp) & 7u;
  unsigned char* myws = p.ws[rank];
  const unsigned lds0 = (unsigned)(uintptr_t)(LAS unsigned char*)smem;

  unsigned* seqp = (unsigned*)(myws + SEQ_OFF) + b;
  const unsigned seq = __hip_atomic_load(seqp, __ATOMIC_RELAXED, __HIP_MEMORY_SCOPE_AGENT) + 1u;
  const u32x4 xv0 = *(const u32x4*)(p.x + (size_t)b * 8192 + tid * 16);
  const u32x4 xv1 = *(const u32x4*)(p.x + (size_t)b * 8192 + 4096 + tid * 16);
  unsigned* myflag = (unsigned*)(p.ws[(unsigned)(rank + lane) & 7u] + FLAG_OFF) + rank * NWG + b;
  const float scale = p.sx[0] * p.sw[0];
  asm volatile("" :: "v"(myflag), "v"(scale), "v"(xv0), "v"(xv1), "v"(seq));
  if (w == 0) asm volatile("buffer_inv sc0 sc1" ::: "memory");
  st16x2_sys(myws + EXP_OFF + (size_t)(seq & 1u) * SHARD_BYTES + (size_t)b * 8192 + tid * 16, xv0, xv1);

  const unsigned char* bptr[4];
#pragma unroll
  for (int i = 0; i < 4; ++i) {
    const int kr = 32 * w + 8 * i + (lane >> 3);
    const int fb = ((kr >> 1) & 3) | (((kr >> 4) & 1) << 2);
    bptr[i] = p.w + (size_t)kr * NS + cb * BN + (((lane & 7) ^ fb) << 4);
  }
  auto stage_b = [&](int ks) {
    const unsigned base = lds0 + (unsigned)(ks % NSLOT) * STAGE + A_BYTES + (unsigned)w * 4096;
#pragma unroll
    for (int i = 0; i < 4; ++i) glds16(bptr[i] + (size_t)ks * (BK * NS), base + i * 1024);
  };
#pragma unroll
  for (int s = 0; s < PD; ++s) stage_b(s);

  WAIT_VM(12);
  __builtin_amdgcn_s_barrier();
  if (w == 0) {
    if (lane >= 1 && lane < NR) __hip_atomic_store(myflag, seq, __ATOMIC_RELAXED, __HIP_MEMORY_SCOPE_SYSTEM);
    unsigned timed_out = 0;
    if (lane == 0) __hip_atomic_store(seqp, seq, __ATOMIC_RELAXED, __HIP_MEMORY_SCOPE_AGENT);
    if (grp != 0) {
      unsigned* f = (unsigned*)(myws + FLAG_OFF) + src * NWG + rb * 32 + (lane & 31);
      unsigned* tmo = (unsigned*)(myws + TMO_OFF);
      const unsigned long long t0 = __builtin_amdgcn_s_memrealtime();
      for (;;) {
        const unsigned v = __hip_atomic_load(f, __ATOMIC_RELAXED, __HIP_MEMORY_SCOPE_SYSTEM);
        if (__all((int)(v - seq) >= 0)) break;
        if (__hip_atomic_load(tmo, __ATOMIC_RELAXED, __HIP_MEMORY_SCOPE_AGENT) == seq ||
            __builtin_amdgcn_s_memrealtime() - t0 > DEADLINE_TICKS) {
          if (lane == 0) __hip_atomic_store(tmo, seq, __ATOMIC_RELAXED, __HIP_MEMORY_SCOPE_AGENT);
          timed_out = 1;
          break;
        }
        __builtin_amdgcn_s_sleep(1);
      }
    }
    if (lane == 0) *(volatile unsigned*)(smem + NSLOT * STAGE) = timed_out;
    asm volatile("s_waitcnt vmcnt(0) lgkmcnt(0)" ::: "memory");
  }
  __builtin_amdgcn_s_barrier();
  const bool bad = *(volatile unsigned*)(smem + NSLOT * STAGE) != 0;

  const unsigned char* abase = (grp == 0) ? p.x : (p.ws[src] + EXP_OFF + (size_t)(seq & 1u) * SHARD_BYTES);
  const unsigned char* aptr[2];
#pragma unroll
  for (int i = 0; i < 2; ++i) {
    const int r = 16 * w + 8 * i + (lane >> 3);
    const int fa = (r >> 1) & 7;
    aptr[i] = abase + (size_t)(rb * BM + r) * KD + (((lane & 7) ^ fa) << 4);
  }
  auto stage_a = [&](int ks) {
    const unsigned base = lds0 + (unsigned)(ks % NSLOT) * STAGE + (unsigned)w * 2048;
#pragma unroll
    for (int i = 0; i < 2; ++i) glds16(aptr[i] + ks * BK, base + i * 1024);
  };
#pragma unroll
  for (int s = 0; s < PD; ++s) stage_a(s);

  const int wm = w >> 1, wn = w & 1, fr = lane & 15, g = lane >> 4;
  unsigned xoff[2][2], woff[4][4];
#pragma unroll
  for (int mf = 0; mf < 2; ++mf)
#pragma unroll
    for (int h = 0; h < 2; ++h) {
      const int r = 32 * wm + 16 * mf + fr;
      xoff[mf][h] = r * 128 + (((g + 4 * h) ^ ((r >> 1) & 7)) << 4);
    }
#pragma unroll
  for (int nf = 0; nf < 4; ++nf)
#pragma unroll
    for (int c = 0; c < 4; ++c) {
      const int kr = 64 * (c >> 1) + 16 * g + 8 * (c & 1) + (fr >> 1);
      const int fb = ((kr >> 1) & 3) | (((kr >> 4) & 1) << 2);
      woff[nf][c] = A_BYTES + kr * 128 + (((4 * wn + nf) ^ fb) << 4) + 8 * (fr & 1);
    }

  v4f acc[4][2];
#pragma unroll
  for (int nf = 0; nf < 4; ++nf)
#pragma unroll
    for (int mf = 0; mf < 2; ++mf) acc[nf][mf] = (v4f){0.f, 0.f, 0.f, 0.f};

  for (int ks = 0; ks < NSTEP; ++ks) {
    if (ks == 0) WAIT_VM(4);
    else if (ks == 1) WAIT_VM(8);
    else if (ks <= NSTEP - PD) WAIT_VM(12);
    else if (ks == NSTEP - 2) WAIT_VM(6);
    else WAIT_VM(0);
    __builtin_amdgcn_s_barrier();
    if (ks + PD < NSTEP) { stage_a(ks + PD); stage_b(ks + PD); }
    const unsigned char* sb = smem + (ks % NSLOT) * STAGE;
    v8i xf[2], wf[4];
#pragma unroll
    for (int mf = 0; mf < 2; ++mf) {
      const v4i lo = *(const v4i*)(sb + xoff[mf][0]);
      const v4i hi = *(const v4i*)(sb + xoff[mf][1]);
      xf[mf] = (v8i){lo.x, lo.y, lo.z, lo.w, hi.x, hi.y, hi.z, hi.w};
    }
#pragma unroll
    for (int nf = 0; nf < 4; ++nf) {
      v2i t[4];
#pragma unroll
      for (int c = 0; c < 4; ++c)
        t[c] = __builtin_amdgcn_ds_read_tr8_b64_v2i32((LAS v2i*)(sb + woff[nf][c]));
      wf[nf] = (v8i){t[0].x, t[0].y, t[1].x, t[1].y, t[2].x, t[2].y, t[3].x, t[3].y};
    }
#pragma unroll
    for (int nf = 0; nf < 4; ++nf)
#pragma unroll
      for (int mf = 0; mf < 2; ++mf)
        acc[nf][mf] = __builtin_amdgcn_mfma_scale_f32_16x16x128_f8f6f4(wf[nf], xf[mf], acc[nf][mf], 1, 0, 0, 127, 0, 127);
    asm volatile("s_waitcnt lgkmcnt(0)" ::: "memory");
  }

  const float nanv = __builtin_nanf("");
#pragma unroll
  for (int mf = 0; mf < 2; ++mf) {
    float* orow = p.out + (size_t)(src * MS + rb * BM + 32 * wm + 16 * mf + fr) * NS + cb * BN + 64 * wn + 4 * g;
#pragma unroll
    for (int nf = 0; nf < 4; ++nf) {
      v4f v = acc[nf][mf] * scale;
      v.x = fmaxf(v.x, 0.f); v.y = fmaxf(v.y, 0.f); v.z = fmaxf(v.z, 0.f); v.w = fmaxf(v.w, 0.f);
      if (bad) v = (v4f){nanv, nanv, nanv, nanv};
      *(v4f*)(orow + 16 * nf) = v;
    }
  }
}

}

extern "C" void kernel_launch(int rank, int nranks, ncclComm_t comm,
                              hipStream_t stream,
                              void* const* d_in, const int* in_sizes, int n_in,
                              void* d_out, int out_size,
                              const int* in_dtypes, int out_dtype,
                              void* const* d_out_all,
                              void* const* d_ws_all, size_t ws_bytes) {
  if (nranks != NR || ws_bytes < WS_NEED || n_in != 4) { fprintf(stderr, "unexpected configuration\n"); return; }
  HIP_CHECK(hipFuncSetAttribute((const void*)ag_gemm_kernel, hipFuncAttributeMaxDynamicSharedMemorySize, LDS_BYTES));
  Params p;
  p.x = (const unsigned char*)d_in[0];
  p.w = (const unsigned char*)d_in[1];
  p.sx = (const float*)d_in[2];
  p.sw = (const float*)d_in[3];
  p.out = (float*)d_out;
  for (int i = 0; i < NR; ++i) p.ws[i] = (unsigned char*)d_ws_all[i];
  p.rank = rank;
  ag_gemm_kernel<<<NWG, 256, LDS_BYTES, stream>>>(p);
  HIP_CHECK(hipGetLastError());
}
```

```cpp
#include <hip/hip_runtime.h>
#include <rccl/rccl.h>
#include <cstdio>
#include <cstdint>

#define HIP_CHECK(expr) do { hipError_t _e = (expr); if (_e != hipSuccess) { \
    fprintf(stderr, "HIP error %s at %s:%d\n", hipGetErrorString(_e), __FILE__, __LINE__); \
    return; } } while (0)

namespace {

typedef int v2i __attribute__((ext_vector_type(2)));
typedef int v4i __attribute__((ext_vector_type(4)));
typedef int v8i __attribute__((ext_vector_type(8)));
typedef float v4f __attribute__((ext_vector_type(4)));
typedef unsigned u32x4 __attribute__((ext_vector_type(4)));
#define LAS __attribute__((address_space(3)))

#define KP 0
#define PD 2

constexpr int NR = 8;
constexpr int MS = 512;
constexpr int KD = 4096;
constexpr int NS = 512;
constexpr int BM = 64, BN = 128, BK = 128, NSTEP = KD / BK;
constexpr int A_BYTES = BM * BK, B_BYTES = BK * BN;
constexpr int LDS_B = 0, LDS_A = 4 * B_BYTES, LDS_MISC = LDS_A + 2 * A_BYTES;
static_assert(PD == 2, "the w ring (distance 4) must run two steps ahead of the x loads");
constexpr int LDS_BYTES = LDS_MISC + 2048;
constexpr int NWG = 256;
constexpr unsigned SEC_RAW = 2048;
constexpr unsigned SEC_CMP = 1840;
constexpr size_t SHARD_BYTES = (size_t)MS * KD;
constexpr size_t EXPR_OFF = 0;
constexpr size_t EXPC_OFF = 2 * SHARD_BYTES;
constexpr size_t STG_OFF = 4 * SHARD_BYTES;
constexpr size_t STG_SRC = (size_t)(KP > 0 ? KP : 1) * 8 * 8192;
constexpr size_t FLAG_OFF = STG_OFF + 2 * NR * STG_SRC;
constexpr size_t PFLAG_OFF = FLAG_OFF + 2 * NR * 4096;
constexpr size_t SEQ_OFF = PFLAG_OFF + 2 * NR * 1024;
constexpr size_t TMO_OFF = SEQ_OFF + 4096;
constexpr size_t WS_NEED = TMO_OFF + 256;
constexpr unsigned long long DEADLINE_TICKS = 100000000ull;

struct Params {
  const unsigned char* x;
  const unsigned char* w;
  const float* sx;
  const float* sw;
  float* out;
  unsigned char* ws[NR];
  int rank;
};

__device__ __forceinline__ void glds16(const void* g, unsigned lds_addr) {
  unsigned keep;
  asm volatile("s_mov_b32 %0, m0\n\ts_mov_b32 m0, %2\n\ts_nop 0\n\tglobal_load_lds_dwordx4 %1, off\n\ts_mov_b32 m0, %0"
               : "=&s"(keep) : "v"(g), "s"(lds_addr) : "memory");
}
__device__ __forceinline__ void st16x2_sys(void* p, u32x4 v0, u32x4 v1) {
  asm volatile("global_store_dwordx4 %0, %1, off sc0 sc1\n\tglobal_store_dwordx4 %0, %2, off offset:1024 sc0 sc1\n\ts_nop 1"
               :: "v"(p), "v"(v0), "v"(v1) : "memory");
}
#define WAIT_VM(n) asm volatile("s_waitcnt vmcnt(%0)" :: "i"(n) : "memory")
#define TS(i) do {} while (0)

__host__ __device__ constexpr int wcount(int i) {
  if (i < PD) return 3 * (PD - 1 - i) + 7 * i;
  int n = 0;
  for (int j = i - PD; j < i; ++j) n += (j > i - PD && j + PD < NSTEP ? 3 : 0) + (j + 4 < NSTEP ? 4 : 0);
  return n;
}
static_assert(wcount(2) == wcount(28), "steady-state wait");

__device__ __forceinline__ unsigned wave_scan(unsigned x) {
  x += __builtin_amdgcn_update_dpp(0u, x, 0x111, 0xf, 0xf, true);
  x += __builtin_amdgcn_update_dpp(0u, x, 0x112, 0xf, 0xf, true);
  x += __builtin_amdgcn_update_dpp(0u, x, 0x114, 0xf, 0xf, true);
  x += __builtin_amdgcn_update_dpp(0u, x, 0x118, 0xf, 0xf, true);
  x += __builtin_amdgcn_update_dpp(0u, x, 0x142, 0xa, 0xf, false);
  x += __builtin_amdgcn_update_dpp(0u, x, 0x143, 0xc, 0xf, false);
  return x;
}
__device__ __forceinline__ unsigned shift_pos(unsigned v, int from, int to) { return to >= from ? v << (to - from) : v >> (from - to); }
__device__ constexpr int CPOS[7] = {0, 1, 2, 3, 4, 5, 7};

__device__ __forceinline__ int kmap(int i) {
  if (KP == 0) return i;
  if (KP == 2) return i < 10 ? i + 2 : (i == 10 ? 0 : (i < 21 ? i + 1 : (i == 21 ? 1 : i)));
  return i < 10 ? i + 4 : (i == 10 ? 0 : (i < 15 ? i + 3 : (i == 15 ? 1 : (i < 20 ? i + 2 : (i == 20 ? 2 : (i < 25 ? i + 1 : (i == 25 ? 3 : i)))))));
}

__global__ __launch_bounds__(256, 1) void ag_gemm_kernel(Params p) {
  extern __shared__ __attribute__((aligned(16))) unsigned char smem[];
  const int tid = threadIdx.x;
  const int lane = tid & 63;
  const int w = __builtin_amdgcn_readfirstlane(tid >> 6);
  const int b = blockIdx.x;
  const int grp = b & 7, idx = b >> 3, rb = idx >> 2, cb = idx & 3;
  const int rank = p.rank;
  const unsigned src = (unsigned)(rank + grp) & 7u;
  unsigned char* myws = p.ws[rank];
  const unsigned lds0 = (unsigned)(uintptr_t)(LAS unsigned char*)smem;
  TS(0);

  const unsigned rowoff = (unsigned)(16 * w + (lane >> 2)) * KD + (lane & 3) * 16;
  const unsigned char* xp = p.x + (size_t)(grp * BM) * KD + idx * BK + rowoff;
  const u32x4 xv0 = *(const u32x4*)xp;
  const u32x4 xv1 = *(const u32x4*)(xp + 64);
  asm volatile("" ::: "memory");
  unsigned* seqp = (unsigned*)(myws + SEQ_OFF) + b * 4 + w;
  const unsigned seq = __hip_atomic_load(seqp, __ATOMIC_RELAXED, __HIP_MEMORY_SCOPE_AGENT) + 1u;
  unsigned char* peerws = p.ws[(unsigned)(rank + lane) & 7u];
  const float scale = p.sx[0] * p.sw[0];
  asm volatile("" :: "v"(peerws), "v"(scale), "v"(xv0), "v"(xv1), "v"(seq));
  const unsigned q = seq & 1u;
  const unsigned tag = (seq & 0xFFFFFFu) << 8;
  unsigned meta = 0;
  if (idx >= KP) {
    unsigned D[8] = {xv0.x, xv0.y, xv0.z, xv0.w, xv1.x, xv1.y, xv1.z, xv1.w};
    LAS unsigned char* scr = (LAS unsigned char*)smem + LDS_A + w * 2048;
    constexpr int NIB = A_BYTES;
    *(LAS u32x4*)(scr + lane * 32) = xv0;
    *(LAS u32x4*)(scr + lane * 32 + 16) = xv1;
    if (lane < 16) *(LAS unsigned*)(scr + NIB + lane * 4) = 0u;
    unsigned base;
    {
      unsigned sum = 0;
#pragma unroll
      for (int i = 0; i < 8; ++i) sum = __builtin_amdgcn_sad_u8(D[i] & 0x78787878u, 0u, sum);
      const int t = ((int)__builtin_amdgcn_readlane(wave_scan(sum), 63) - 54067) >> 14;
      base = t < 0 ? 0u : (t > 8 ? 8u : (unsigned)t);
    }
    const unsigned cB = (128u - 8u * base) * 0x01010101u, cB7 = (72u - 8u * base) * 0x01010101u;
    unsigned C[8], M = 0;
#pragma unroll
    for (int i = 0; i < 8; ++i) {
      const unsigned u = D[i] & 0x78787878u;
      const unsigned g1 = u + cB, g2 = u + cB7;
      const unsigned esc = (~g1 | g2) & 0x80808080u;
      C[i] = (D[i] & 0x87878787u) | (g1 & 0x38383838u) | ((esc - (esc >> 3)) >> 1);
      M |= esc >> (7 - i);
    }
#pragma unroll
    for (int i = 0; i < 7; ++i) C[i] |= shift_pos(C[7], CPOS[i], 6) & 0x40404040u;
    const unsigned cnt = __builtin_popcount(M);
    const unsigned inc = wave_scan(cnt);
    const unsigned total = __builtin_amdgcn_readlane(inc, 63);
    if (total <= 96u && !__any(cnt > 8u)) {
      unsigned S = 0, Mm = M;
      unsigned by[8];
#pragma unroll
      for (int k = 0; k < 8; ++k) {
        const unsigned bt = (unsigned)__builtin_ctz(Mm | 0x80000000u);
        by[k] = *(LAS unsigned char*)(scr + lane * 32 + ((bt & 7) << 2) + (bt >> 3));
        Mm &= Mm - 1;
      }
#pragma unroll
      for (int k = 0; k < 8; ++k) S |= (k < (int)cnt ? (by[k] >> 3) & 15u : 0u) << (4 * k);
      const unsigned pfx = inc - cnt, sh = (pfx & 7u) * 4u;
      if (cnt) {
        LAS unsigned* sp = (LAS unsigned*)(scr + NIB) + (pfx >> 3);
        __hip_atomic_fetch_or(sp, S << sh, __ATOMIC_RELAXED, __HIP_MEMORY_SCOPE_WORKGROUP);
        __hip_atomic_fetch_or(sp + 1, (unsigned)(((unsigned long long)S << sh) >> 32), __ATOMIC_RELAXED, __HIP_MEMORY_SCOPE_WORKGROUP);
      }
      const unsigned stv = *(volatile LAS unsigned*)(scr + NIB + (lane < 12 ? lane : 11) * 4);
      unsigned char* sec = myws + EXPC_OFF + (size_t)q * SHARD_BYTES + (size_t)(b * 4 + w) * SEC_CMP;
      typedef unsigned u32x3 __attribute__((ext_vector_type(3)));
      const u32x4 c0 = {C[0], C[1], C[2], C[3]};
      const u32x3 c1 = {C[4], C[5], C[6]};
      asm volatile("global_store_dwordx4 %0, %3, off sc0 sc1\n\tglobal_store_dwordx3 %1, %4, off sc0 sc1\n\t"
                   "global_store_dword %2, %5, off sc0 sc1\n\ts_nop 1"
                   :: "v"(sec + lane * 16), "v"(sec + 1024 + lane * 12), "v"(sec + 1792 + (lane < 12 ? lane : 11) * 4),
                      "v"(c0), "v"(c1), "v"(stv) : "memory");
      meta = 0x80u | base;
    } else {
      st16x2_sys(myws + EXPR_OFF + (size_t)q * SHARD_BYTES + (size_t)(b * 4 + w) * SEC_RAW + lane * 16, xv0, xv1);
    }
  }

  const unsigned char* bptr[4];
#pragma unroll
  for (int i = 0; i < 4; ++i) {
    const int kr = 32 * w + 8 * i + (lane >> 3);
    const int fb = ((kr >> 1) & 3) | (((kr >> 4) & 1) << 2);
    bptr[i] = p.w + (size_t)kr * NS + cb * BN + (((lane & 7) ^ fb) << 4);
  }
  auto stage_b = [&](int pos) __attribute__((always_inline)) {
    const unsigned base = lds0 + LDS_B + (unsigned)(pos & 3) * B_BYTES + (unsigned)w * 4096;
    const size_t go = (size_t)kmap(pos) * (BK * NS);
#pragma unroll
    for (int i = 0; i < 4; ++i) glds16(bptr[i] + go, base + i * 1024);
  };
#pragma unroll
  for (int s = 0; s < 4; ++s) stage_b(s);
  TS(1);
  WAIT_VM(16 + KP / 2);
  if (lane >= 1 && lane < NR)
    __hip_atomic_store((unsigned*)(peerws + FLAG_OFF) + ((q * NR + rank) * 8 + grp) * 128 + w * 32 + idx, tag | meta,
                       __ATOMIC_RELAXED, __HIP_MEMORY_SCOPE_SYSTEM);
  if (lane == 0) __hip_atomic_store(seqp, seq, __ATOMIC_RELAXED, __HIP_MEMORY_SCOPE_AGENT);
  TS(2);

  unsigned* tmo = (unsigned*)(myws + TMO_OFF);
  const unsigned* fptr = lane < 32 ? (const unsigned*)(myws + FLAG_OFF) + ((q * NR + src) * 8 + rb) * 128 + w * 32 + lane
                       : lane < 40 ? (const unsigned*)(myws + PFLAG_OFF) + (q * NR + src) * NWG + rb * 32 + w * 8 + (lane - 32) : tmo;
  unsigned flags = tag;
  unsigned bad = 0;
  auto poll = [&](int last) __attribute__((always_inline)) {
    const unsigned long long t0 = __builtin_amdgcn_s_memrealtime();
    for (;;) {
      flags = __hip_atomic_load(fptr, __ATOMIC_RELAXED, __HIP_MEMORY_SCOPE_SYSTEM);
      const bool ok = lane < KP || lane >= last || (flags >> 8) == (tag >> 8);
      if (__all(ok)) break;
      if (__builtin_amdgcn_readlane(flags, 40) == seq || __builtin_amdgcn_s_memrealtime() - t0 > DEADLINE_TICKS) {
        if (lane == 0) __hip_atomic_store(tmo, seq, __ATOMIC_RELAXED, __HIP_MEMORY_SCOPE_AGENT);
        bad = 1;
        break;
      }
      __builtin_amdgcn_s_sleep(1);
    }
  };
  if (grp != 0) poll(32);
  TS(3);

  const unsigned o1 = grp == 0 ? rowoff : (unsigned)lane * 16;
  const unsigned o2 = grp == 0 ? rowoff + 64 : 1024u + lane * 16;
  const unsigned char* own_base = p.x + (size_t)(rb * BM) * KD;
  const unsigned char* exp_base = p.ws[src] + EXPR_OFF + (size_t)q * SHARD_BYTES + (size_t)(rb * 4 + w) * SEC_RAW;
  const unsigned char* cmp_base = p.ws[src] + EXPC_OFF + (size_t)q * SHARD_BYTES + (size_t)(rb * 4 + w) * SEC_CMP;
  const unsigned o2c = 1024u + lane * 12, o3c = 1792u + (lane < 12 ? lane : 11) * 4;
  const unsigned char* stg_base = myws + STG_OFF + ((size_t)q * NR + src) * STG_SRC + (size_t)(rb * 4 + w) * SEC_RAW;
#define LOAD_A(LO, HI, ST, ME, POS) do {                                                                         \
    const int ks_ = kmap(POS);                                                                                   \
    ME = (grp != 0 && ks_ >= KP) ? (__builtin_amdgcn_readlane(flags, ks_) & 0xFFu) : 0u;                        \
    if (KP > 0 && grp != 0 && ks_ < KP) {                                                                        \
      const unsigned char* base_ = stg_base + (size_t)ks_ * (8 * 8192);                                          \
      asm volatile("global_load_dwordx4 %0, %3, %5 sc0 sc1\n\tglobal_load_dwordx4 %1, %4, %5 sc0 sc1\n\t"       \
                   "global_load_dword %2, %3, %5 sc0 sc1"                                                        \
                   : "=&v"(LO), "=&v"(HI), "=&v"(ST) : "v"(o1), "v"(o2), "s"(base_) : "memory");                 \
    } else {                                                                                                     \
      const bool c_ = (ME & 0x80u) != 0;                                                                         \
      const unsigned char* base_ = c_ ? cmp_base + (size_t)ks_ * (32 * SEC_CMP)                                  \
                                      : (grp == 0 ? own_base + ks_ * BK : exp_base + (size_t)ks_ * (8 * 8192));  \
      const unsigned oa_ = c_ ? o2c : o2, ob_ = c_ ? o3c : o1;                                                   \
      asm volatile("global_load_dwordx4 %0, %3, %6\n\tglobal_load_dwordx4 %1, %4, %6\n\tglobal_load_dword %2, %5, %6" \
                   : "=&v"(LO), "=&v"(HI), "=&v"(ST) : "v"(o1), "v"(oa_), "v"(ob_), "s"(base_) : "memory");      \
    }                                                                                                            \
  } while (0)
  u32x4 lo0, hi0, lo1, hi1;
  unsigned st0, st1, me0, me1;
  LOAD_A(lo0, hi0, st0, me0, 0);
  LOAD_A(lo1, hi1, st1, me1, 1);

  const int wm = w >> 1, wn = w & 1, fr = lane & 15, g = lane >> 4;
  unsigned awr0, awr1;
  {
    const int r = 16 * w + (lane >> 2), fa = (r >> 1) & 7;
    awr0 = r * 128 + (((lane & 3) ^ fa) << 4);
    awr1 = r * 128 + ((((lane & 3) + 4) ^ fa) << 4);
  }
  const unsigned xo0 = (32 * wm + fr) * 128 + ((g ^ ((fr >> 1) & 7)) << 4);
  const unsigned xo1 = (32 * wm + fr) * 128 + (((g + 4) ^ ((fr >> 1) & 7)) << 4);
  unsigned wo[4];
#pragma unroll
  for (int nf = 0; nf < 4; ++nf)
    wo[nf] = (16 * g + (fr >> 1)) * 128 + (((4 * wn + nf) ^ (((fr >> 2) & 3) | ((g & 1) << 2))) << 4) + 8 * (fr & 1);

  v4f acc[4][2];
#pragma unroll
  for (int nf = 0; nf < 4; ++nf)
#pragma unroll
    for (int mf = 0; mf < 2; ++mf) acc[nf][mf] = (v4f){0.f, 0.f, 0.f, 0.f};

  v8i wf[4];
#define READ_W(POS) do {                                                                                         \
    const unsigned char* sb_ = smem + LDS_B + ((POS) & 3) * B_BYTES;                                             \
    _Pragma("unroll") for (int nf = 0; nf < 4; ++nf) {                                                           \
      v2i t_[4];                                                                                                 \
      _Pragma("unroll") for (int c = 0; c < 4; ++c)                                                              \
        t_[c] = __builtin_amdgcn_ds_read_tr8_b64_v2i32((LAS v2i*)(sb_ + wo[nf] + (64 * (c >> 1) + 8 * (c & 1)) * 128)); \
      wf[nf] = (v8i){t_[0].x, t_[0].y, t_[1].x, t_[1].y, t_[2].x, t_[2].y, t_[3].x, t_[3].y};                    \
    }                                                                                                            \
  } while (0)
#define STEP(LO, HI, ST, ME, I, U) do {                                                                          \
    const int i = (I);                                                                                           \
                                          \
    if (i == 0) WAIT_VM(wcount(0));                                                                              \
    else if (i == 1) WAIT_VM(wcount(1));                                                                         \
    else if (i <= 28) WAIT_VM(wcount(28));                                                                       \
    else if (i == 29) WAIT_VM(wcount(29));                                                                       \
    else if (i == 30) WAIT_VM(wcount(30));                                                                       \
    else WAIT_VM(0);                                                                                             \
    asm volatile("" : "+v"(LO), "+v"(HI), "+v"(ST));                                                             \
    TS(4 + i);                                                                                                   \
    unsigned char* sa = smem + LDS_A + (U) * A_BYTES;                                                            \
    if (ME & 0x80u) {                                                                                            \
      const unsigned base = ME & 15u;                                                                            \
      unsigned C[8] = {LO.x, LO.y, LO.z, LO.w, HI.x, HI.y, HI.z, 0u};                                            \
      _Pragma("unroll") for (int k = 0; k < 7; ++k) C[7] |= shift_pos(C[k] & 0x40404040u, 6, CPOS[k]);           \
      unsigned M = 0, Dd[8];                                                                                     \
      _Pragma("unroll") for (int k = 0; k < 8; ++k) {                                                            \
        const unsigned t = C[k] & 0xBFBFBFBFu;                                                                   \
        M |= shift_pos((t + 0x08080808u) & 0x40404040u, 6, k);                                                   \
        Dd[k] = t + base * 0x08080808u;                                                                          \
      }                                                                                                          \
      *(u32x4*)(sa + awr0) = (u32x4){Dd[0], Dd[1], Dd[2], Dd[3]};                                                \
      *(u32x4*)(sa + awr1) = (u32x4){Dd[4], Dd[5], Dd[6], Dd[7]};                                                \
      const unsigned cnt = __builtin_popcount(M);                                                                \
      const unsigned pfx = wave_scan(cnt) - cnt;                                                                 \
      const unsigned w0 = __builtin_amdgcn_ds_bpermute((int)((pfx >> 3) << 2), (int)ST);                         \
      const unsigned w1 = __builtin_amdgcn_ds_bpermute((int)(((pfx >> 3) + 1) << 2), (int)ST);                   \
      unsigned S = __builtin_amdgcn_alignbit(w1, w0, (pfx & 7u) * 4u);                                           \
      while (M) {                                                                                                \
        const unsigned bt = (unsigned)__builtin_ctz(M);                                                          \
        M &= M - 1;                                                                                              \
        const unsigned a = ((bt & 4u) ? awr1 : awr0) + ((bt & 3u) << 2);                                         \
        const unsigned dv = ((S & 15u) - base - 7u) << ((bt & 24u) + 3u);                                        \
        S >>= 4;                                                                                                 \
        __hip_atomic_fetch_add((LAS unsigned*)(sa + a), dv, __ATOMIC_RELAXED, __HIP_MEMORY_SCOPE_WORKGROUP);     \
      }                                                                                                          \
    } else {                                                                                                     \
      *(u32x4*)(sa + awr0) = LO;                                                                                 \
      *(u32x4*)(sa + awr1) = HI;                                                                                 \
    }                                                                                                            \
    if (KP > 0 && grp != 0) {                                                                                    \
      if (i == 3) {                            \
        unsigned keep;                                                                                           \
        asm volatile("s_mov_b32 %0, m0\n\ts_mov_b32 m0, %2\n\ts_nop 0\n\tglobal_load_lds_dword %1, off sc0 sc1\n\ts_mov_b32 m0, %0" \
                     : "=&s"(keep) : "v"(fptr), "s"(lds0 + LDS_MISC + 256 + 256 * w) : "memory");                \
      } else if (i == 4 + PD) {                                                     \
        const unsigned f2 = *(const volatile LAS unsigned*)(smem + LDS_MISC + 256 + 256 * w + 4 * lane);         \
        if (!__all(lane < KP || lane >= 40 || (f2 >> 8) == (tag >> 8)) && !bad) poll(40);                        \
      }                                                                                                          \
    }                                                                                                            \
    asm volatile("s_waitcnt lgkmcnt(0)" ::: "memory");                          \
    if (i + PD < NSTEP) LOAD_A(LO, HI, ST, ME, i + PD);                                                          \
    if (i == NSTEP - 1) {                                                                                        \
      if (lane == 0) *(volatile LAS unsigned*)(smem + LDS_MISC + 4 * w) = bad;                                   \
      asm volatile("s_waitcnt lgkmcnt(0)" ::: "memory");                                                         \
    }                                                                                                            \
    __builtin_amdgcn_s_barrier();                                                                                \
    if (i + 4 < NSTEP) stage_b(i + 4);                                                                           \
    v8i xf[2];                                                                                                   \
    _Pragma("unroll") for (int mf = 0; mf < 2; ++mf) {                                                           \
      const v4i l_ = *(const v4i*)(sa + xo0 + mf * 2048);                                                        \
      const v4i h_ = *(const v4i*)(sa + xo1 + mf * 2048);                                                        \
      xf[mf] = (v8i){l_.x, l_.y, l_.z, l_.w, h_.x, h_.y, h_.z, h_.w};                                            \
    }                                                                                                            \
    _Pragma("unroll") for (int nf = 0; nf < 4; ++nf)                                                             \
      _Pragma("unroll") for (int mf = 0; mf < 2; ++mf)                                                           \
        acc[nf][mf] = __builtin_amdgcn_mfma_scale_f32_16x16x128_f8f6f4(wf[nf], xf[mf], acc[nf][mf], 1, 0, 0, 127, 0, 127); \
                                         \
    asm volatile("" : "+a"(acc[0][0]), "+a"(acc[0][1]), "+a"(acc[1][0]), "+a"(acc[1][1]),                        \
                      "+a"(acc[2][0]), "+a"(acc[2][1]), "+a"(acc[3][0]), "+a"(acc[3][1]));                       \
    if (i + 1 < NSTEP) READ_W(i + 1);                                                                            \
  } while (0)
  if (grp == 0) WAIT_VM(3 * PD);
  __builtin_amdgcn_s_barrier();
  READ_W(0);
#pragma unroll 1
  for (int ii = 0; ii < NSTEP; ii += 2) {
    STEP(lo0, hi0, st0, me0, ii, 0);
    STEP(lo1, hi1, st1, me1, ii + 1, 1);
  }

  const u32x4 badv = *(const u32x4*)(smem + LDS_MISC);
  const bool anybad = (badv.x | badv.y | badv.z | badv.w) != 0;
  const float sc = anybad ? __builtin_nanf("") : scale;
#pragma unroll
  for (int mf = 0; mf < 2; ++mf) {
    float* orow = p.out + (size_t)(src * MS + rb * BM + 32 * wm + 16 * mf + fr) * NS + cb * BN + 64 * wn + 4 * g;
#pragma unroll
    for (int nf = 0; nf < 4; ++nf) {
      v4f v = acc[nf][mf] * sc;
      v.x = v.x < 0.f ? 0.f : v.x; v.y = v.y < 0.f ? 0.f : v.y; v.z = v.z < 0.f ? 0.f : v.z; v.w = v.w < 0.f ? 0.f : v.w;
      *(v4f*)(orow + 16 * nf) = v;
    }
  }
}

}

extern "C" void kernel_launch(int rank, int nranks, ncclComm_t comm,
                              hipStream_t stream,
                              void* const* d_in, const int* in_sizes, int n_in,
                              void* d_out, int out_size,
                              const int* in_dtypes, int out_dtype,
                              void* const* d_out_all,
                              void* const* d_ws_all, size_t ws_bytes) {
  if (nranks != NR || ws_bytes < WS_NEED || n_in != 4) { fprintf(stderr, "unexpected configuration\n"); return; }
  HIP_CHECK(hipFuncSetAttribute((const void*)ag_gemm_kernel, hipFuncAttributeMaxDynamicSharedMemorySize, LDS_BYTES));
  Params p;
  p.x = (const unsigned char*)d_in[0];
  p.w = (const unsigned char*)d_in[1];
  p.sx = (const float*)d_in[2];
  p.sw = (const float*)d_in[3];
  p.out = (float*)d_out;
  for (int i = 0; i < NR; ++i) p.ws[i] = (unsigned char*)d_ws_all[i];
  p.rank = rank;
  ag_gemm_kernel<<<NWG, 256, LDS_BYTES, stream>>>(p);
  HIP_CHECK(hipGetLastError());
}
```

```cpp
#include <hip/hip_runtime.h>
#include <rccl/rccl.h>
#include <cstdio>
#include <cstdint>

#define HIP_CHECK(expr) do { hipError_t _e = (expr); if (_e != hipSuccess) { \
    fprintf(stderr, "HIP error %s at %s:%d\n", hipGetErrorString(_e), __FILE__, __LINE__); \
    return; } } while (0)

namespace {

typedef int v2i __attribute__((ext_vector_type(2)));
typedef int v4i __attribute__((ext_vector_type(4)));
typedef int v8i __attribute__((ext_vector_type(8)));
typedef float v4f __attribute__((ext_vector_type(4)));
typedef unsigned u32x4 __attribute__((ext_vector_type(4)));
#define LAS __attribute__((address_space(3)))

#define OST 2
#define IM 2
#define RAWK 0
#define PD 2

constexpr int NR = 8;
constexpr int MS = 512;
constexpr int KD = 4096;
constexpr int NS = 512;
constexpr int BM = 64, BN = 128, BK = 128, NSTEP = KD / BK;
constexpr int A_BYTES = BM * BK, B_BYTES = BK * BN;
constexpr int LDS_B = 0, LDS_A = 4 * B_BYTES, LDS_MISC = LDS_A + 2 * A_BYTES;
static_assert(RAWK == 0, "");
static_assert(PD == 2, "the w ring (distance 4) must run two steps ahead of the x loads");
constexpr int LDS_BYTES = LDS_MISC + 1024;
constexpr int NWG = 256;
constexpr int NTHR = 256;
constexpr unsigned SEC_RAW = 2048;
constexpr unsigned SEC_CMP = 1840;
constexpr unsigned BLK_CMP = 7424;
constexpr size_t SHARD_BYTES = (size_t)MS * KD;
constexpr size_t EXPR_OFF = 0;
constexpr size_t EXPC_OFF = 2 * SHARD_BYTES;
constexpr size_t FLAG_OFF = 4 * SHARD_BYTES;
constexpr size_t SEQ_OFF = FLAG_OFF + 2 * NR * 4096;
constexpr size_t TMO_OFF = SEQ_OFF + 8192;
constexpr size_t WS_NEED = TMO_OFF + 256;
constexpr unsigned long long DEADLINE_TICKS = 100000000ull;

struct Params {
  const unsigned char* x;
  const unsigned char* w;
  const float* sx;
  const float* sw;
  float* out;
  unsigned char* ws[NR];
  int rank;
};

__device__ __forceinline__ void glds16(const void* g, unsigned lds_addr) {
  unsigned keep;
  asm volatile("s_mov_b32 %0, m0\n\ts_mov_b32 m0, %2\n\ts_nop 0\n\tglobal_load_lds_dwordx4 %1, off\n\ts_mov_b32 m0, %0"
               : "=&s"(keep) : "v"(g), "s"(lds_addr) : "memory");
}
__device__ __forceinline__ void st16x2_sys(void* p, u32x4 v0, u32x4 v1) {
  asm volatile("global_store_dwordx4 %0, %1, off sc0 sc1\n\tglobal_store_dwordx4 %0, %2, off offset:1024 sc0 sc1\n\ts_nop 1"
               :: "v"(p), "v"(v0), "v"(v1) : "memory");
}
#define WAIT_VM(n) asm volatile("s_waitcnt vmcnt(%0)" :: "i"(n) : "memory")
#define TS(i) do {} while (0)

__host__ __device__ constexpr int wcount(int i) {
  if (i < PD) return 3 * (PD - 1 - i) + 7 * i;
  int n = 0;
  for (int j = i - PD; j < i; ++j) n += (j > i - PD && j + PD < NSTEP ? 3 : 0) + (j + 4 < NSTEP && (IM < 2 || j > i - PD) ? 4 : 0);
  return n;
}
static_assert(wcount(2) == wcount(27), "steady-state wait");

__device__ __forceinline__ unsigned wave_scan(unsigned x) {
  x += __builtin_amdgcn_update_dpp(0u, x, 0x111, 0xf, 0xf, true);
  x += __builtin_amdgcn_update_dpp(0u, x, 0x112, 0xf, 0xf, true);
  x += __builtin_amdgcn_update_dpp(0u, x, 0x114, 0xf, 0xf, true);
  x += __builtin_amdgcn_update_dpp(0u, x, 0x118, 0xf, 0xf, true);
  x += __builtin_amdgcn_update_dpp(0u, x, 0x142, 0xa, 0xf, false);
  x += __builtin_amdgcn_update_dpp(0u, x, 0x143, 0xc, 0xf, false);
  return x;
}
__device__ __forceinline__ unsigned shift_pos(unsigned v, int from, int to) { return to >= from ? v << (to - from) : v >> (from - to); }
__device__ constexpr int CPOS[7] = {0, 1, 2, 3, 4, 5, 7};

__global__ __launch_bounds__(NTHR, 1) void ag_gemm_kernel(Params p) {
  extern __shared__ __attribute__((aligned(16))) unsigned char smem[];
  const int tid = threadIdx.x;
  const int lane = tid & 63;
  const int w = __builtin_amdgcn_readfirstlane(tid >> 6);
  const int b = blockIdx.x;
  const int grp = b & 7, idx = b >> 3, rb = idx >> 2, cb = idx & 3;
  const int rank = p.rank;
  const unsigned src = (unsigned)(rank + grp) & 7u;
  unsigned char* myws = p.ws[rank];
  const unsigned lds0 = (unsigned)(uintptr_t)(LAS unsigned char*)smem;
  TS(0);

  const unsigned rowoff = (unsigned)(16 * w + (lane >> 2)) * KD + (lane & 3) * 16;
  const unsigned char* xp = p.x + (size_t)(grp * BM) * KD + idx * BK + rowoff;
  unsigned* seqp = (unsigned*)(myws + SEQ_OFF) + b;
  unsigned char* peerws = p.ws[(unsigned)(rank + lane) & 7u];
  asm volatile("" :: "v"(peerws));
  u32x4 xv0, xv1;
  unsigned seq;
  float sxv, swv;
  asm volatile("global_load_dwordx4 %0, %5, off\n\tglobal_load_dwordx4 %1, %5, off offset:64\n\tglobal_load_dword %2, %6, off sc1\n\t"
               "global_load_dword %3, %7, off\n\tglobal_load_dword %4, %8, off"
               : "=&v"(xv0), "=&v"(xv1), "=&v"(seq), "=&v"(sxv), "=&v"(swv) : "v"(xp), "v"(seqp), "v"(p.sx), "v"(p.sw) : "memory");
  const unsigned char* bptr[4];
#pragma unroll
  for (int i = 0; i < 4; ++i) {
    const int kr = 32 * w + 8 * i + (lane >> 3);
    const int fb = ((kr >> 1) & 3) | (((kr >> 4) & 1) << 2);
    bptr[i] = p.w + (size_t)kr * NS + cb * BN + (((lane & 7) ^ fb) << 4);
  }
  auto stage_b = [&](int pos) __attribute__((always_inline)) {
    const unsigned base = lds0 + LDS_B + (unsigned)(pos & 3) * B_BYTES + (unsigned)w * 4096;
    const size_t go = (size_t)pos * (BK * NS);
#pragma unroll
    for (int i = 0; i < 4; ++i) glds16(bptr[i] + go, base + i * 1024);
  };
#pragma unroll
  for (int s = 0; s < 4; ++s) stage_b(s);
  asm volatile("s_waitcnt vmcnt(16)" : "+v"(xv0), "+v"(xv1), "+v"(seq), "+v"(sxv), "+v"(swv) :: "memory");
  seq = __builtin_amdgcn_readfirstlane(seq) + 1u;
  const float scale = sxv * swv;
  const unsigned q = seq & 1u;
  const unsigned tag = (seq & 0xFFFFFFu) << 8;
  unsigned meta = 0;
  if (idx < RAWK) {
    st16x2_sys(myws + EXPR_OFF + (size_t)q * SHARD_BYTES + (size_t)(b * 4 + w) * SEC_RAW + lane * 16, xv0, xv1);
  } else {
    unsigned D[8] = {xv0.x, xv0.y, xv0.z, xv0.w, xv1.x, xv1.y, xv1.z, xv1.w};
    LAS unsigned char* scr = (LAS unsigned char*)smem + LDS_A + w * 2048;
    constexpr int NIB = A_BYTES;
    *(LAS u32x4*)(scr + lane * 32) = xv0;
    *(LAS u32x4*)(scr + lane * 32 + 16) = xv1;
    if (lane < 16) *(LAS unsigned*)(scr + NIB + lane * 4) = 0u;
    unsigned base;
    {
      unsigned sum = 0;
#pragma unroll
      for (int i = 0; i < 8; ++i) sum = __builtin_amdgcn_sad_u8(D[i] & 0x78787878u, 0u, sum);
      const int t = ((int)__builtin_amdgcn_readlane(wave_scan(sum), 63) - 54067) >> 14;
      base = t < 0 ? 0u : (t > 8 ? 8u : (unsigned)t);
    }
    const unsigned cB = (128u - 8u * base) * 0x01010101u, cB7 = (72u - 8u * base) * 0x01010101u;
    unsigned C[8], M = 0;
#pragma unroll
    for (int i = 0; i < 8; ++i) {
      const unsigned u = D[i] & 0x78787878u;
      const unsigned g1 = u + cB, g2 = u + cB7;
      const unsigned esc = (~g1 | g2) & 0x80808080u;
      C[i] = (D[i] & 0x87878787u) | (g1 & 0x38383838u) | ((esc - (esc >> 3)) >> 1);
      M |= esc >> (7 - i);
    }
#pragma unroll
    for (int i = 0; i < 7; ++i) C[i] |= shift_pos(C[7], CPOS[i], 6) & 0x40404040u;
    const unsigned cnt = __builtin_popcount(M);
    const unsigned inc = wave_scan(cnt);
    const unsigned total = __builtin_amdgcn_readlane(inc, 63);
    if (total <= 96u && !__any(cnt > 8u)) {
      unsigned S = 0, Mm = M;
      unsigned by[8];
#pragma unroll
      for (int k = 0; k < 8; ++k) {
        const unsigned bt = (unsigned)__builtin_ctz(Mm | 0x80000000u);
        by[k] = *(LAS unsigned char*)(scr + lane * 32 + ((bt & 7) << 2) + (bt >> 3));
        Mm &= Mm - 1;
      }
#pragma unroll
      for (int k = 0; k < 8; ++k) S |= (k < (int)cnt ? (by[k] >> 3) & 15u : 0u) << (4 * k);
      const unsigned pfx = inc - cnt, sh = (pfx & 7u) * 4u;
      if (cnt) {
        LAS unsigned* sp = (LAS unsigned*)(scr + NIB) + (pfx >> 3);
        __hip_atomic_fetch_or(sp, S << sh, __ATOMIC_RELAXED, __HIP_MEMORY_SCOPE_WORKGROUP);
        __hip_atomic_fetch_or(sp + 1, (unsigned)(((unsigned long long)S << sh) >> 32), __ATOMIC_RELAXED, __HIP_MEMORY_SCOPE_WORKGROUP);
      }
      const unsigned stv = *(volatile LAS unsigned*)(scr + NIB + (lane < 12 ? lane : 11) * 4);
      unsigned char* sec = myws + EXPC_OFF + (size_t)q * SHARD_BYTES + (size_t)b * BLK_CMP + w * SEC_CMP;
      typedef unsigned u32x3 __attribute__((ext_vector_type(3)));
      const u32x4 c0 = {C[0], C[1], C[2], C[3]};
      const u32x3 c1 = {C[4], C[5], C[6]};
      asm volatile("global_store_dwordx4 %0, %3, off sc0 sc1\n\tglobal_store_dwordx3 %1, %4, off sc0 sc1\n\t"
                   "global_store_dword %2, %5, off sc0 sc1\n\ts_nop 1"
                   :: "v"(sec + lane * 16), "v"(sec + 1024 + lane * 12), "v"(sec + 1792 + (lane < 12 ? lane : 11) * 4),
                      "v"(c0), "v"(c1), "v"(stv) : "memory");
      meta = 0x80u | base;
    } else {
      st16x2_sys(myws + EXPR_OFF + (size_t)q * SHARD_BYTES + (size_t)(b * 4 + w) * SEC_RAW + lane * 16, xv0, xv1);
    }
  }
  TS(1);
  LAS unsigned* misc = (LAS unsigned*)(smem + LDS_MISC);
  WAIT_VM(0);
  if (lane == 0) misc[w] = tag | meta;
  asm volatile("s_waitcnt lgkmcnt(0)" ::: "memory");
  __builtin_amdgcn_s_barrier();
  TS(2);
  unsigned* tmo = (unsigned*)(myws + TMO_OFF);
  if (w == 3) {
    const u32x4 fv = *(const LAS u32x4*)misc;
    if (lane >= 1 && lane < NR)
      asm volatile("global_store_dwordx4 %0, %1, off sc0 sc1\n\ts_nop 1"
                   :: "v"((unsigned*)(peerws + FLAG_OFF) + ((q * NR + rank) * 8 + grp) * 128 + idx * 4), "v"(fv) : "memory");
    if (lane == 0) __hip_atomic_store(seqp, seq, __ATOMIC_RELAXED, __HIP_MEMORY_SCOPE_AGENT);
  } else if (w == 0) {
    unsigned bad = 0;
    u32x4 f = {0u, 0u, 0u, 0u};
    if (grp != 0) {
      const unsigned* fptr = lane < 32 ? (const unsigned*)(myws + FLAG_OFF) + ((q * NR + src) * 8 + rb) * 128 + lane * 4 : tmo;
      const unsigned long long t0 = __builtin_amdgcn_s_memrealtime();
      for (;;) {
        asm volatile("global_load_dwordx4 %0, %1, off sc0 sc1\n\ts_waitcnt vmcnt(0)" : "=v"(f) : "v"(fptr) : "memory");
        if (__all(lane >= 32 || (((f.x ^ tag) | (f.y ^ tag) | (f.z ^ tag) | (f.w ^ tag)) >> 8) == 0)) break;
        if (__builtin_amdgcn_readlane(f.x, 32) == seq || __builtin_amdgcn_s_memrealtime() - t0 > DEADLINE_TICKS) {
          if (lane == 0) __hip_atomic_store(tmo, seq, __ATOMIC_RELAXED, __HIP_MEMORY_SCOPE_AGENT);
          bad = 1;
          f = (u32x4){0u, 0u, 0u, 0u};
          break;
        }
      }
    }
    if (lane < 32) *(LAS u32x4*)(misc + 64 + lane * 4) = f;
    if (lane == 0) misc[4] = bad;
    asm volatile("s_waitcnt lgkmcnt(0)" ::: "memory");
  }
  __builtin_amdgcn_s_barrier();
  const unsigned flags = misc[64 + (lane & 31) * 4 + w];
  const bool anybad = misc[4] != 0;
  TS(3);

  const unsigned o1 = grp == 0 ? rowoff : (unsigned)lane * 16;
  const unsigned o2 = grp == 0 ? rowoff + 64 : 1024u + lane * 16;
  const unsigned char* own_base = p.x + (size_t)(rb * BM) * KD;
  const unsigned char* exp_base = p.ws[src] + EXPR_OFF + (size_t)q * SHARD_BYTES + (size_t)(rb * 4 + w) * SEC_RAW;
  const unsigned char* cmp_base = p.ws[src] + EXPC_OFF + (size_t)q * SHARD_BYTES + (size_t)rb * BLK_CMP + w * SEC_CMP;
  const unsigned o2c = 1024u + lane * 12, o3c = 1792u + (lane < 12 ? lane : 11) * 4;
#define LOAD_A(LO, HI, ST, ME, POS) do {                                                                         \
    const int ks_ = (POS);                                                                                       \
    ME = __builtin_amdgcn_readlane(flags, ks_) & 0xFFu;                                                          \
    {                                                                                                            \
      const bool c_ = (ME & 0x80u) != 0;                                                                         \
      const unsigned char* base_ = c_ ? cmp_base + (size_t)ks_ * (8 * BLK_CMP)                                   \
                                      : (grp == 0 ? own_base + ks_ * BK : exp_base + (size_t)ks_ * (8 * 8192));  \
      const unsigned oa_ = c_ ? o2c : o2, ob_ = c_ ? o3c : o1;                                                   \
      asm volatile("global_load_dwordx4 %0, %3, %6\n\tglobal_load_dwordx4 %1, %4, %6\n\tglobal_load_dword %2, %5, %6" \
                   : "=&v"(LO), "=&v"(HI), "=&v"(ST) : "v"(o1), "v"(oa_), "v"(ob_), "s"(base_) : "memory");      \
    }                                                                                                            \
  } while (0)
  u32x4 lo0, hi0, lo1, hi1;
  unsigned st0, st1, me0, me1;
  LOAD_A(lo0, hi0, st0, me0, 0);
  LOAD_A(lo1, hi1, st1, me1, 1);

  const int wm = w >> 1, wn = w & 1, fr = lane & 15, g = lane >> 4;
  unsigned awr0, awr1;
  {
    const int r = 16 * w + (lane >> 2), fa = (r >> 1) & 7;
    awr0 = r * 128 + (((lane & 3) ^ fa) << 4);
    awr1 = r * 128 + ((((lane & 3) + 4) ^ fa) << 4);
  }
  const unsigned xo0 = (32 * wm + fr) * 128 + ((g ^ ((fr >> 1) & 7)) << 4);
  const unsigned xo1 = (32 * wm + fr) * 128 + (((g + 4) ^ ((fr >> 1) & 7)) << 4);
  unsigned wo[4];
#pragma unroll
  for (int nf = 0; nf < 4; ++nf)
    wo[nf] = (16 * g + (fr >> 1)) * 128 + (((4 * wn + nf) ^ (((fr >> 2) & 3) | ((g & 1) << 2))) << 4) + 8 * (fr & 1);

  v4f acc[4][2];
#pragma unroll
  for (int nf = 0; nf < 4; ++nf)
#pragma unroll
    for (int mf = 0; mf < 2; ++mf) acc[nf][mf] = (v4f){0.f, 0.f, 0.f, 0.f};

  v8i wf[4];
#define READ_W(POS) do {                                                                                         \
    const unsigned char* sb_ = smem + LDS_B + ((POS) & 3) * B_BYTES;                                             \
    _Pragma("unroll") for (int nf = 0; nf < 4; ++nf) {                                                           \
      v2i t_[4];                                                                                                 \
      _Pragma("unroll") for (int c = 0; c < 4; ++c)                                                              \
        t_[c] = __builtin_amdgcn_ds_read_tr8_b64_v2i32((LAS v2i*)(sb_ + wo[nf] + (64 * (c >> 1) + 8 * (c & 1)) * 128)); \
      wf[nf] = (v8i){t_[0].x, t_[0].y, t_[1].x, t_[1].y, t_[2].x, t_[2].y, t_[3].x, t_[3].y};                    \
    }                                                                                                            \
  } while (0)
#define STEP(LO, HI, ST, ME, I, U) do {                                                                          \
    const int i = (I);                                                                                           \
    if (i == 16) TS(44);                                                                                         \
                                          \
    if (i == 0) WAIT_VM(wcount(0));                                                                              \
    else if (i == 1) WAIT_VM(wcount(1));                                                                         \
    else if (i <= 27) WAIT_VM(wcount(27));                                                                       \
    else if (i == 28) WAIT_VM(wcount(28));                                                                       \
    else if (i == 29) WAIT_VM(wcount(29));                                                                       \
    else if (i == 30) WAIT_VM(wcount(30));                                                                       \
    else WAIT_VM(0);                                                                                             \
    asm volatile("" : "+v"(LO), "+v"(HI), "+v"(ST));                                                             \
    TS(4 + i);                                                                                                   \
    unsigned char* sa = smem + LDS_A + (U) * A_BYTES;                                                            \
    u32x4 lo_ = LO, hi_ = HI; unsigned st_ = ST; const unsigned me_ = ME;                                        \
    if (IM == 1) {                                                                                               \
      asm volatile("" : "+v"(lo_), "+v"(hi_), "+v"(st_));                                                        \
      if (i + PD < NSTEP) LOAD_A(LO, HI, ST, ME, i + PD);                                                        \
    }                                                                                                            \
    if (me_ & 0x80u) {                                                                                           \
      const unsigned base = me_ & 15u;                                                                           \
      unsigned C[8] = {lo_.x, lo_.y, lo_.z, lo_.w, hi_.x, hi_.y, hi_.z, 0u};                                     \
      _Pragma("unroll") for (int k = 0; k < 7; ++k) C[7] |= shift_pos(C[k] & 0x40404040u, 6, CPOS[k]);           \
      unsigned M = 0, Dd[8];                                                                                     \
      _Pragma("unroll") for (int k = 0; k < 8; ++k) {                                                            \
        const unsigned t = C[k] & 0xBFBFBFBFu;                                                                   \
        M |= shift_pos((t + 0x08080808u) & 0x40404040u, 6, k);                                                   \
        Dd[k] = t + base * 0x08080808u;                                                                          \
      }                                                                                                          \
      *(u32x4*)(sa + awr0) = (u32x4){Dd[0], Dd[1], Dd[2], Dd[3]};                                                \
      *(u32x4*)(sa + awr1) = (u32x4){Dd[4], Dd[5], Dd[6], Dd[7]};                                                \
      const unsigned cnt = __builtin_popcount(M);                                                                \
      const unsigned pfx = wave_scan(cnt) - cnt;                                                                 \
      const unsigned w0 = __builtin_amdgcn_ds_bpermute((int)((pfx >> 3) << 2), (int)st_);                        \
      const unsigned w1 = __builtin_amdgcn_ds_bpermute((int)(((pfx >> 3) + 1) << 2), (int)st_);                  \
      unsigned S = __builtin_amdgcn_alignbit(w1, w0, (pfx & 7u) * 4u);                                           \
      while (M) {                                                                                                \
        const unsigned bt = (unsigned)__builtin_ctz(M);                                                          \
        M &= M - 1;                                                                                              \
        const unsigned a = ((bt & 4u) ? awr1 : awr0) + ((bt & 3u) << 2);                                         \
        const unsigned dv = ((S & 15u) - base - 7u) << ((bt & 24u) + 3u);                                        \
        S >>= 4;                                                                                                 \
        __hip_atomic_fetch_add((LAS unsigned*)(sa + a), dv, __ATOMIC_RELAXED, __HIP_MEMORY_SCOPE_WORKGROUP);     \
      }                                                                                                          \
    } else {                                                                                                     \
      *(u32x4*)(sa + awr0) = lo_;                                                                                \
      *(u32x4*)(sa + awr1) = hi_;                                                                                \
    }                                                                                                            \
    asm volatile("s_waitcnt lgkmcnt(0)" ::: "memory");                          \
    if (IM == 0 && i + PD < NSTEP) LOAD_A(LO, HI, ST, ME, i + PD);                                               \
    if (i == 16) TS(45);                                                                                         \
    __builtin_amdgcn_s_barrier();                                                                                \
    if (i == 16) TS(46);                                                                                         \
    if (i + 4 < NSTEP) stage_b(i + 4);                                                                           \
    if (IM == 2 && i + PD < NSTEP) LOAD_A(LO, HI, ST, ME, i + PD);                                               \
    v8i xf[2];                                                                                                   \
    _Pragma("unroll") for (int mf = 0; mf < 2; ++mf) {                                                           \
      const v4i l_ = *(const v4i*)(sa + xo0 + mf * 2048);                                                        \
      const v4i h_ = *(const v4i*)(sa + xo1 + mf * 2048);                                                        \
      xf[mf] = (v8i){l_.x, l_.y, l_.z, l_.w, h_.x, h_.y, h_.z, h_.w};                                            \
    }                                                                                                            \
    _Pragma("unroll") for (int nf = 0; nf < 4; ++nf)                                                             \
      _Pragma("unroll") for (int mf = 0; mf < 2; ++mf)                                                           \
        acc[nf][mf] = __builtin_amdgcn_mfma_scale_f32_16x16x128_f8f6f4(wf[nf], xf[mf], acc[nf][mf], 1, 0, 0, 127, 0, 127); \
                                         \
    asm volatile("" : "+a"(acc[0][0]), "+a"(acc[0][1]), "+a"(acc[1][0]), "+a"(acc[1][1]),                        \
                      "+a"(acc[2][0]), "+a"(acc[2][1]), "+a"(acc[3][0]), "+a"(acc[3][1]));                       \
    if (i == 16) TS(47);                                                                                         \
    if (i + 1 < NSTEP) READ_W(i + 1);                                                                            \
    if (IM == 3 && i + PD < NSTEP) LOAD_A(LO, HI, ST, ME, i + PD);                                               \
  } while (0)
  READ_W(0);
#pragma unroll 1
  for (int ii = 0; ii < NSTEP; ii += 2) {
    STEP(lo0, hi0, st0, me0, ii, 0);
    STEP(lo1, hi1, st1, me1, ii + 1, 1);
  }

  const float sc = anybad ? __builtin_nanf("") : scale;
#pragma unroll
  for (int mf = 0; mf < 2; ++mf) {
    float* orow = p.out + (size_t)(src * MS + rb * BM + 32 * wm + 16 * mf + fr) * NS + cb * BN + 64 * wn + 4 * g;
#pragma unroll
    for (int nf = 0; nf < 4; ++nf) {
      v4f v = acc[nf][mf] * sc;
      v.x = v.x < 0.f ? 0.f : v.x; v.y = v.y < 0.f ? 0.f : v.y; v.z = v.z < 0.f ? 0.f : v.z; v.w = v.w < 0.f ? 0.f : v.w;
      asm volatile("global_store_dwordx4 %0, %1, off sc1\n\ts_nop 1" :: "v"(orow + 16 * nf), "v"(v) : "memory");
    }
  }
}

}

extern "C" void kernel_launch(int rank, int nranks, ncclComm_t comm,
                              hipStream_t stream,
                              void* const* d_in, const int* in_sizes, int n_in,
                              void* d_out, int out_size,
                              const int* in_dtypes, int out_dtype,
                              void* const* d_out_all,
                              void* const* d_ws_all, size_t ws_bytes) {
  if (nranks != NR || ws_bytes < WS_NEED || n_in != 4) { fprintf(stderr, "unexpected configuration\n"); return; }
  HIP_CHECK(hipFuncSetAttribute((const void*)ag_gemm_kernel, hipFuncAttributeMaxDynamicSharedMemorySize, LDS_BYTES));
  Params p;
  p.x = (const unsigned char*)d_in[0];
  p.w = (const unsigned char*)d_in[1];
  p.sx = (const float*)d_in[2];
  p.sw = (const float*)d_in[3];
  p.out = (float*)d_out;
  for (int i = 0; i < NR; ++i) p.ws[i] = (unsigned char*)d_ws_all[i];
  p.rank = rank;
  ag_gemm_kernel<<<NWG, NTHR, LDS_BYTES, stream>>>(p);
  HIP_CHECK(hipGetLastError());
}
```

```cpp
#include <hip/hip_runtime.h>
#include <rccl/rccl.h>
#include <cstdio>
#include <cstdint>

#define HIP_CHECK(expr) do { hipError_t _e = (expr); if (_e != hipSuccess) { \
    fprintf(stderr, "HIP error %s at %s:%d\n", hipGetErrorString(_e), __FILE__, __LINE__); \
    return; } } while (0)

namespace {

typedef int v2i __attribute__((ext_vector_type(2)));
typedef int v4i __attribute__((ext_vector_type(4)));
typedef int v8i __attribute__((ext_vector_type(8)));
typedef float v4f __attribute__((ext_vector_type(4)));
typedef unsigned u32x4 __attribute__((ext_vector_type(4)));
#define LAS __attribute__((address_space(3)))

#define OST 2
#define IM 2
#define RAWK 1
#define PD 2

constexpr int NR = 8;
constexpr int MS = 512;
constexpr int KD = 4096;
constexpr int NS = 512;
constexpr int BM = 64, BN = 128, BK = 128, NSTEP = KD / BK;
constexpr int A_BYTES = BM * BK, B_BYTES = BK * BN;
constexpr int LDS_B = 0, LDS_A = 4 * B_BYTES, LDS_MISC = LDS_A + 2 * A_BYTES;
static_assert(PD == 2, "the w ring (distance 4) must run two steps ahead of the x loads");
constexpr int LDS_BYTES = LDS_MISC + 2048;
constexpr int NWG = 256;
constexpr int NTHR = RAWK > 0 ? 320 : 256;
constexpr unsigned SEC_RAW = 2048;
constexpr unsigned SEC_CMP = 1840;
constexpr unsigned BLK_CMP = 7424;
constexpr size_t SHARD_BYTES = (size_t)MS * KD;
constexpr size_t EXPR_OFF = 0;
constexpr size_t EXPC_OFF = 2 * SHARD_BYTES;
constexpr size_t FLAG_OFF = 4 * SHARD_BYTES;
constexpr size_t SEQ_OFF = FLAG_OFF + 2 * NR * 4096;
constexpr size_t TMO_OFF = SEQ_OFF + 8192;
constexpr size_t WS_NEED = TMO_OFF + 256;
constexpr unsigned long long DEADLINE_TICKS = 100000000ull;

struct Params {
  const unsigned char* x;
  const unsigned char* w;
  const float* sx;
  const float* sw;
  float* out;
  unsigned char* ws[NR];
  int rank;
};

__device__ __forceinline__ void glds16(const void* g, unsigned lds_addr) {
  unsigned keep;
  asm volatile("s_mov_b32 %0, m0\n\ts_mov_b32 m0, %2\n\ts_nop 0\n\tglobal_load_lds_dwordx4 %1, off\n\ts_mov_b32 m0, %0"
               : "=&s"(keep) : "v"(g), "s"(lds_addr) : "memory");
}
__device__ __forceinline__ void st16x2_sys(void* p, u32x4 v0, u32x4 v1) {
  asm volatile("global_store_dwordx4 %0, %1, off sc0 sc1\n\tglobal_store_dwordx4 %0, %2, off offset:1024 sc0 sc1\n\ts_nop 1"
               :: "v"(p), "v"(v0), "v"(v1) : "memory");
}
#define WAIT_VM(n) asm volatile("s_waitcnt vmcnt(%0)" :: "i"(n) : "memory")
#define TS(i) do {} while (0)

__host__ __device__ constexpr int wcount(int i) {
  if (i < PD) return 3 * (PD - 1 - i) + 7 * i;
  int n = 0;
  for (int j = i - PD; j < i; ++j) n += (j > i - PD && j + PD < NSTEP ? 3 : 0) + (j + 4 < NSTEP && (IM < 2 || j > i - PD) ? 4 : 0);
  return n;
}
static_assert(wcount(2) == wcount(27), "steady-state wait");

__device__ __forceinline__ unsigned wave_scan(unsigned x) {
  x += __builtin_amdgcn_update_dpp(0u, x, 0x111, 0xf, 0xf, true);
  x += __builtin_amdgcn_update_dpp(0u, x, 0x112, 0xf, 0xf, true);
  x += __builtin_amdgcn_update_dpp(0u, x, 0x114, 0xf, 0xf, true);
  x += __builtin_amdgcn_update_dpp(0u, x, 0x118, 0xf, 0xf, true);
  x += __builtin_amdgcn_update_dpp(0u, x, 0x142, 0xa, 0xf, false);
  x += __builtin_amdgcn_update_dpp(0u, x, 0x143, 0xc, 0xf, false);
  return x;
}
__device__ __forceinline__ unsigned shift_pos(unsigned v, int from, int to) { return to >= from ? v << (to - from) : v >> (from - to); }
__device__ constexpr int CPOS[7] = {0, 1, 2, 3, 4, 5, 7};

__global__ __launch_bounds__(NTHR, 1) void ag_gemm_kernel(Params p) {
  extern __shared__ __attribute__((aligned(16))) unsigned char smem[];
  const int tid = threadIdx.x;
  const int lane = tid & 63;
  const int w = __builtin_amdgcn_readfirstlane(tid >> 6);
  const int b = blockIdx.x;
  const int grp = b & 7, idx = b >> 3, rb = idx >> 2, cb = idx & 3;
  const int rank = p.rank;
  const unsigned src = (unsigned)(rank + grp) & 7u;
  unsigned char* myws = p.ws[rank];
  const unsigned lds0 = (unsigned)(uintptr_t)(LAS unsigned char*)smem;
  TS(0);

  LAS unsigned* misc = (LAS unsigned*)(smem + LDS_MISC);
  unsigned* seqp = (unsigned*)(myws + SEQ_OFF) + b;
  unsigned* tmo = (unsigned*)(myws + TMO_OFF);
  if (w == 4) {
    const unsigned seq = __hip_atomic_load(seqp, __ATOMIC_RELAXED, __HIP_MEMORY_SCOPE_AGENT) + 1u;
    if (lane < 4) misc[lane] = 0u;
    asm volatile("s_waitcnt vmcnt(0) lgkmcnt(0)" :: "v"(seq) : "memory");
    __builtin_amdgcn_s_barrier();
    if (grp != 0) {
      const unsigned q = seq & 1u, tag = (seq & 0xFFFFFFu) << 8;
      const unsigned* fptr = (const unsigned*)(myws + FLAG_OFF) + ((q * NR + src) * 8 + rb) * 128 + (lane < RAWK ? lane : 0) * 4;
      const unsigned long long t0 = __builtin_amdgcn_s_memrealtime();
      for (unsigned it = 1;; ++it) {
        u32x4 f;
        asm volatile("global_load_dwordx4 %0, %1, off sc0 sc1\n\ts_waitcnt vmcnt(0)" : "=v"(f) : "v"(fptr) : "memory");
        if (__all(lane >= RAWK || (((f.x ^ tag) | (f.y ^ tag) | (f.z ^ tag) | (f.w ^ tag)) >> 8) == 0)) {
          const unsigned char* g = p.ws[src] + EXPR_OFF + (size_t)q * SHARD_BYTES + (size_t)(rb * 4 + cb) * SEC_RAW + lane * 16;
#pragma unroll
          for (int k = 0; k < RAWK; ++k) {
            glds16(g + (size_t)k * (8 * 8192), lds0 + LDS_MISC + 1024);
            glds16(g + (size_t)k * (8 * 8192) + 1024, lds0 + LDS_MISC + 1024);
          }
          break;
        }
        if ((it & 63u) == 0) {
          if (__hip_atomic_load(tmo, __ATOMIC_RELAXED, __HIP_MEMORY_SCOPE_AGENT) == seq ||
              __builtin_amdgcn_s_memrealtime() - t0 > DEADLINE_TICKS) break;
        }
        if (it > 64u) __builtin_amdgcn_s_sleep(8);
      }
    }
    return;
  }

  const unsigned rowoff = (unsigned)(16 * w + (lane >> 2)) * KD + (lane & 3) * 16;
  const unsigned char* xp = p.x + (size_t)(grp * BM) * KD + idx * BK + rowoff;
  unsigned char* peerws = p.ws[(unsigned)(rank + lane) & 7u];
  asm volatile("" :: "v"(peerws));
  u32x4 xv0, xv1;
  unsigned seq;
  float sxv, swv;
  asm volatile("global_load_dwordx4 %0, %5, off\n\tglobal_load_dwordx4 %1, %5, off offset:64\n\tglobal_load_dword %2, %6, off sc1\n\t"
               "global_load_dword %3, %7, off\n\tglobal_load_dword %4, %8, off"
               : "=&v"(xv0), "=&v"(xv1), "=&v"(seq), "=&v"(sxv), "=&v"(swv) : "v"(xp), "v"(seqp), "v"(p.sx), "v"(p.sw) : "memory");
  const unsigned char* bptr[4];
#pragma unroll
  for (int i = 0; i < 4; ++i) {
    const int kr = 32 * w + 8 * i + (lane >> 3);
    const int fb = ((kr >> 1) & 3) | (((kr >> 4) & 1) << 2);
    bptr[i] = p.w + (size_t)kr * NS + cb * BN + (((lane & 7) ^ fb) << 4);
  }
  auto stage_b = [&](int pos) __attribute__((always_inline)) {
    const unsigned base = lds0 + LDS_B + (unsigned)(pos & 3) * B_BYTES + (unsigned)w * 4096;
    const size_t go = (size_t)pos * (BK * NS);
#pragma unroll
    for (int i = 0; i < 4; ++i) glds16(bptr[i] + go, base + i * 1024);
  };
#pragma unroll
  for (int s = 0; s < 4; ++s) stage_b(s);
  asm volatile("s_waitcnt vmcnt(16)" : "+v"(xv0), "+v"(xv1), "+v"(seq), "+v"(sxv), "+v"(swv) :: "memory");
  __builtin_amdgcn_s_barrier();
  seq = __builtin_amdgcn_readfirstlane(seq) + 1u;
  const float scale = sxv * swv;
  const unsigned q = seq & 1u;
  const unsigned tag = (seq & 0xFFFFFFu) << 8;
  unsigned meta = 0;
  if (idx < RAWK) {
    st16x2_sys(myws + EXPR_OFF + (size_t)q * SHARD_BYTES + (size_t)(b * 4 + w) * SEC_RAW + lane * 16, xv0, xv1);
  } else {
    unsigned D[8] = {xv0.x, xv0.y, xv0.z, xv0.w, xv1.x, xv1.y, xv1.z, xv1.w};
    LAS unsigned char* scr = (LAS unsigned char*)smem + LDS_A + w * 2048;
    constexpr int NIB = A_BYTES;
    *(LAS u32x4*)(scr + lane * 32) = xv0;
    *(LAS u32x4*)(scr + lane * 32 + 16) = xv1;
    if (lane < 16) *(LAS unsigned*)(scr + NIB + lane * 4) = 0u;
    unsigned base;
    {
      unsigned sum = 0;
#pragma unroll
      for (int i = 0; i < 8; ++i) sum = __builtin_amdgcn_sad_u8(D[i] & 0x78787878u, 0u, sum);
      const int t = ((int)__builtin_amdgcn_readlane(wave_scan(sum), 63) - 54067) >> 14;
      base = t < 0 ? 0u : (t > 8 ? 8u : (unsigned)t);
    }
    const unsigned cB = (128u - 8u * base) * 0x01010101u, cB7 = (72u - 8u * base) * 0x01010101u;
    unsigned C[8], M = 0;
#pragma unroll
    for (int i = 0; i < 8; ++i) {
      const unsigned u = D[i] & 0x78787878u;
      const unsigned g1 = u + cB, g2 = u + cB7;
      const unsigned esc = (~g1 | g2) & 0x80808080u;
      C[i] = (D[i] & 0x87878787u) | (g1 & 0x38383838u) | ((esc - (esc >> 3)) >> 1);
      M |= esc >> (7 - i);
    }
#pragma unroll
    for (int i = 0; i < 7; ++i) C[i] |= shift_pos(C[7], CPOS[i], 6) & 0x40404040u;
    const unsigned cnt = __builtin_popcount(M);
    const unsigned inc = wave_scan(cnt);
    const unsigned total = __builtin_amdgcn_readlane(inc, 63);
    if (total <= 96u && !__any(cnt > 8u)) {
      unsigned S = 0, Mm = M;
      unsigned by[8];
#pragma unroll
      for (int k = 0; k < 8; ++k) {
        const unsigned bt = (unsigned)__builtin_ctz(Mm | 0x80000000u);
        by[k] = *(LAS unsigned char*)(scr + lane * 32 + ((bt & 7) << 2) + (bt >> 3));
        Mm &= Mm - 1;
      }
#pragma unroll
      for (int k = 0; k < 8; ++k) S |= (k < (int)cnt ? (by[k] >> 3) & 15u : 0u) << (4 * k);
      const unsigned pfx = inc - cnt, sh = (pfx & 7u) * 4u;
      if (cnt) {
        LAS unsigned* sp = (LAS unsigned*)(scr + NIB) + (pfx >> 3);
        __hip_atomic_fetch_or(sp, S << sh, __ATOMIC_RELAXED, __HIP_MEMORY_SCOPE_WORKGROUP);
        __hip_atomic_fetch_or(sp + 1, (unsigned)(((unsigned long long)S << sh) >> 32), __ATOMIC_RELAXED, __HIP_MEMORY_SCOPE_WORKGROUP);
      }
      const unsigned stv = *(volatile LAS unsigned*)(scr + NIB + (lane < 12 ? lane : 11) * 4);
      unsigned char* sec = myws + EXPC_OFF + (size_t)q * SHARD_BYTES + (size_t)b * BLK_CMP + w * SEC_CMP;
      typedef unsigned u32x3 __attribute__((ext_vector_type(3)));
      const u32x4 c0 = {C[0], C[1], C[2], C[3]};
      const u32x3 c1 = {C[4], C[5], C[6]};
      asm volatile("global_store_dwordx4 %0, %3, off sc0 sc1\n\tglobal_store_dwordx3 %1, %4, off sc0 sc1\n\t"
                   "global_store_dword %2, %5, off sc0 sc1\n\ts_nop 1"
                   :: "v"(sec + lane * 16), "v"(sec + 1024 + lane * 12), "v"(sec + 1792 + (lane < 12 ? lane : 11) * 4),
                      "v"(c0), "v"(c1), "v"(stv) : "memory");
      meta = 0x80u | base;
    } else {
      st16x2_sys(myws + EXPR_OFF + (size_t)q * SHARD_BYTES + (size_t)(b * 4 + w) * SEC_RAW + lane * 16, xv0, xv1);
    }
  }
  TS(1);
  WAIT_VM(0);
  if (lane == 0) *(volatile LAS unsigned*)(misc + w) = tag | meta;
  TS(2);
  if (w == 3) {
    u32x4 fv;
    do {
      asm volatile("ds_read_b128 %0, %1\n\ts_waitcnt lgkmcnt(0)" : "=v"(fv) : "v"(lds0 + LDS_MISC) : "memory");
    } while (fv.x == 0u || fv.y == 0u || fv.z == 0u || fv.w == 0u);
    if (lane >= 1 && lane < NR)
      asm volatile("global_store_dwordx4 %0, %1, off sc0 sc1\n\ts_nop 1"
                   :: "v"((unsigned*)(peerws + FLAG_OFF) + ((q * NR + rank) * 8 + grp) * 128 + idx * 4), "v"(fv) : "memory");
    if (lane == 0) __hip_atomic_store(seqp, seq, __ATOMIC_RELAXED, __HIP_MEMORY_SCOPE_AGENT);
  } else if (w == 0) {
    unsigned bad = 0;
    u32x4 f = {0u, 0u, 0u, 0u};
    if (grp != 0) {
      const unsigned* fptr = (const unsigned*)(myws + FLAG_OFF) + ((q * NR + src) * 8 + rb) * 128 + (lane & 31) * 4;
      const unsigned long long t0 = __builtin_amdgcn_s_memrealtime();
      for (unsigned it = 1;; ++it) {
        asm volatile("global_load_dwordx4 %0, %1, off sc0 sc1\n\ts_waitcnt vmcnt(0)" : "=v"(f) : "v"(fptr) : "memory");
        if (__all((((f.x ^ tag) | (f.y ^ tag) | (f.z ^ tag) | (f.w ^ tag)) >> 8) == 0)) break;
        if (it > 64u) __builtin_amdgcn_s_sleep(8);
        if ((it & 63u) == 0 && (__hip_atomic_load(tmo, __ATOMIC_RELAXED, __HIP_MEMORY_SCOPE_AGENT) == seq ||
                                __builtin_amdgcn_s_memrealtime() - t0 > DEADLINE_TICKS)) {
          if (lane == 0) __hip_atomic_store(tmo, seq, __ATOMIC_RELAXED, __HIP_MEMORY_SCOPE_AGENT);
          bad = 1;
          f = (u32x4){0u, 0u, 0u, 0u};
          break;
        }
      }
    }
    if (lane < 32) *(LAS u32x4*)(misc + 64 + lane * 4) = f;
    if (lane == 0) misc[4] = bad;
    asm volatile("s_waitcnt lgkmcnt(0)" ::: "memory");
  }
  __builtin_amdgcn_s_barrier();
  const unsigned flags = misc[64 + (lane & 31) * 4 + w];
  const bool anybad = misc[4] != 0;
  TS(3);

  const unsigned o1 = grp == 0 ? rowoff : (unsigned)lane * 16;
  const unsigned o2 = grp == 0 ? rowoff + 64 : 1024u + lane * 16;
  const unsigned char* own_base = p.x + (size_t)(rb * BM) * KD;
  const unsigned char* exp_base = p.ws[src] + EXPR_OFF + (size_t)q * SHARD_BYTES + (size_t)(rb * 4 + w) * SEC_RAW;
  const unsigned char* cmp_base = p.ws[src] + EXPC_OFF + (size_t)q * SHARD_BYTES + (size_t)rb * BLK_CMP + w * SEC_CMP;
  const unsigned o2c = 1024u + lane * 12, o3c = 1792u + (lane < 12 ? lane : 11) * 4;
#define LOAD_A(LO, HI, ST, ME, POS) do {                                                                         \
    const int ks_ = (POS);                                                                                       \
    ME = __builtin_amdgcn_readlane(flags, ks_) & 0xFFu;                                                          \
    {                                                                                                            \
      const bool c_ = (ME & 0x80u) != 0;                                                                         \
      const unsigned char* base_ = c_ ? cmp_base + (size_t)ks_ * (8 * BLK_CMP)                                   \
                                      : (grp == 0 ? own_base + ks_ * BK : exp_base + (size_t)ks_ * (8 * 8192));  \
      const unsigned oa_ = c_ ? o2c : o2, ob_ = c_ ? o3c : o1;                                                   \
      asm volatile("global_load_dwordx4 %0, %3, %6\n\tglobal_load_dwordx4 %1, %4, %6\n\tglobal_load_dword %2, %5, %6" \
                   : "=&v"(LO), "=&v"(HI), "=&v"(ST) : "v"(o1), "v"(oa_), "v"(ob_), "s"(base_) : "memory");      \
    }                                                                                                            \
  } while (0)
  u32x4 lo0, hi0, lo1, hi1;
  unsigned st0, st1, me0, me1;
  LOAD_A(lo0, hi0, st0, me0, 0);
  LOAD_A(lo1, hi1, st1, me1, 1);

  const int wm = w >> 1, wn = w & 1, fr = lane & 15, g = lane >> 4;
  unsigned awr0, awr1;
  {
    const int r = 16 * w + (lane >> 2), fa = (r >> 1) & 7;
    awr0 = r * 128 + (((lane & 3) ^ fa) << 4);
    awr1 = r * 128 + ((((lane & 3) + 4) ^ fa) << 4);
  }
  const unsigned xo0 = (32 * wm + fr) * 128 + ((g ^ ((fr >> 1) & 7)) << 4);
  const unsigned xo1 = (32 * wm + fr) * 128 + (((g + 4) ^ ((fr >> 1) & 7)) << 4);
  unsigned wo[4];
#pragma unroll
  for (int nf = 0; nf < 4; ++nf)
    wo[nf] = (16 * g + (fr >> 1)) * 128 + (((4 * wn + nf) ^ (((fr >> 2) & 3) | ((g & 1) << 2))) << 4) + 8 * (fr & 1);

  v4f acc[4][2];
#pragma unroll
  for (int nf = 0; nf < 4; ++nf)
#pragma unroll
    for (int mf = 0; mf < 2; ++mf) acc[nf][mf] = (v4f){0.f, 0.f, 0.f, 0.f};

  v8i wf[4];
#define READ_W(POS) do {                                                                                         \
    const unsigned char* sb_ = smem + LDS_B + ((POS) & 3) * B_BYTES;                                             \
    _Pragma("unroll") for (int nf = 0; nf < 4; ++nf) {                                                           \
      v2i t_[4];                                                                                                 \
      _Pragma("unroll") for (int c = 0; c < 4; ++c)                                                              \
        t_[c] = __builtin_amdgcn_ds_read_tr8_b64_v2i32((LAS v2i*)(sb_ + wo[nf] + (64 * (c >> 1) + 8 * (c & 1)) * 128)); \
      wf[nf] = (v8i){t_[0].x, t_[0].y, t_[1].x, t_[1].y, t_[2].x, t_[2].y, t_[3].x, t_[3].y};                    \
    }                                                                                                            \
  } while (0)
#define STEP(LO, HI, ST, ME, I, U) do {                                                                          \
    const int i = (I);                                                                                           \
    if (i == 16) TS(44);                                                                                         \
                                          \
    if (i == 0) WAIT_VM(wcount(0));                                                                              \
    else if (i == 1) WAIT_VM(wcount(1));                                                                         \
    else if (i <= 27) WAIT_VM(wcount(27));                                                                       \
    else if (i == 28) WAIT_VM(wcount(28));                                                                       \
    else if (i == 29) WAIT_VM(wcount(29));                                                                       \
    else if (i == 30) WAIT_VM(wcount(30));                                                                       \
    else WAIT_VM(0);                                                                                             \
    asm volatile("" : "+v"(LO), "+v"(HI), "+v"(ST));                                                             \
    TS(4 + i);                                                                                                   \
    unsigned char* sa = smem + LDS_A + (U) * A_BYTES;                                                            \
    u32x4 lo_ = LO, hi_ = HI; unsigned st_ = ST; const unsigned me_ = ME;                                        \
    if (IM == 1) {                                                                                               \
      asm volatile("" : "+v"(lo_), "+v"(hi_), "+v"(st_));                                                        \
      if (i + PD < NSTEP) LOAD_A(LO, HI, ST, ME, i + PD);                                                        \
    }                                                                                                            \
    if (me_ & 0x80u) {                                                                                           \
      const unsigned base = me_ & 15u;                                                                           \
      unsigned C[8] = {lo_.x, lo_.y, lo_.z, lo_.w, hi_.x, hi_.y, hi_.z, 0u};                                     \
      _Pragma("unroll") for (int k = 0; k < 7; ++k) C[7] |= shift_pos(C[k] & 0x40404040u, 6, CPOS[k]);           \
      unsigned M = 0, Dd[8];                                                                                     \
      _Pragma("unroll") for (int k = 0; k < 8; ++k) {                                                            \
        const unsigned t = C[k] & 0xBFBFBFBFu;                                                                   \
        M |= shift_pos((t + 0x08080808u) & 0x40404040u, 6, k);                                                   \
        Dd[k] = t + base * 0x08080808u;                                                                          \
      }                                                                                                          \
      *(u32x4*)(sa + awr0) = (u32x4){Dd[0], Dd[1], Dd[2], Dd[3]};                                                \
      *(u32x4*)(sa + awr1) = (u32x4){Dd[4], Dd[5], Dd[6], Dd[7]};                                                \
      const unsigned cnt = __builtin_popcount(M);                                                                \
      const unsigned pfx = wave_scan(cnt) - cnt;                                                                 \
      const unsigned w0 = __builtin_amdgcn_ds_bpermute((int)((pfx >> 3) << 2), (int)st_);                        \
      const unsigned w1 = __builtin_amdgcn_ds_bpermute((int)(((pfx >> 3) + 1) << 2), (int)st_);                  \
      unsigned S = __builtin_amdgcn_alignbit(w1, w0, (pfx & 7u) * 4u);                                           \
      while (M) {                                                                                                \
        const unsigned bt = (unsigned)__builtin_ctz(M);                                                          \
        M &= M - 1;                                                                                              \
        const unsigned a = ((bt & 4u) ? awr1 : awr0) + ((bt & 3u) << 2);                                         \
        const unsigned dv = ((S & 15u) - base - 7u) << ((bt & 24u) + 3u);                                        \
        S >>= 4;                                                                                                 \
        __hip_atomic_fetch_add((LAS unsigned*)(sa + a), dv, __ATOMIC_RELAXED, __HIP_MEMORY_SCOPE_WORKGROUP);     \
      }                                                                                                          \
    } else {                                                                                                     \
      *(u32x4*)(sa + awr0) = lo_;                                                                                \
      *(u32x4*)(sa + awr1) = hi_;                                                                                \
    }                                                                                                            \
    asm volatile("s_waitcnt lgkmcnt(0)" ::: "memory");                          \
    if (IM == 0 && i + PD < NSTEP) LOAD_A(LO, HI, ST, ME, i + PD);                                               \
    if (i == 16) TS(45);                                                                                         \
    __builtin_amdgcn_s_barrier();                                                                                \
    if (i == 16) TS(46);                                                                                         \
    if (i + 4 < NSTEP) stage_b(i + 4);                                                                           \
    if (IM == 2 && i + PD < NSTEP) LOAD_A(LO, HI, ST, ME, i + PD);                                               \
    v8i xf[2];                                                                                                   \
    _Pragma("unroll") for (int mf = 0; mf < 2; ++mf) {                                                           \
      const v4i l_ = *(const v4i*)(sa + xo0 + mf * 2048);                                                        \
      const v4i h_ = *(const v4i*)(sa + xo1 + mf * 2048);                                                        \
      xf[mf] = (v8i){l_.x, l_.y, l_.z, l_.w, h_.x, h_.y, h_.z, h_.w};                                            \
    }                                                                                                            \
    _Pragma("unroll") for (int nf = 0; nf < 4; ++nf)                                                             \
      _Pragma("unroll") for (int mf = 0; mf < 2; ++mf)                                                           \
        acc[nf][mf] = __builtin_amdgcn_mfma_scale_f32_16x16x128_f8f6f4(wf[nf], xf[mf], acc[nf][mf], 1, 0, 0, 127, 0, 127); \
                                         \
    asm volatile("" : "+v"(acc[0][0]), "+v"(acc[0][1]), "+v"(acc[1][0]), "+v"(acc[1][1]),                        \
                      "+v"(acc[2][0]), "+v"(acc[2][1]), "+v"(acc[3][0]), "+v"(acc[3][1]));                       \
    if (i == 16) TS(47);                                                                                         \
    if (i + 1 < NSTEP) READ_W(i + 1);                                                                            \
    if (IM == 3 && i + PD < NSTEP) LOAD_A(LO, HI, ST, ME, i + PD);                                               \
  } while (0)
  float* orow = p.out + (size_t)(src * MS + rb * BM + 32 * wm + fr) * NS + cb * BN + 64 * wn + 4 * g;
  asm volatile("" : "+v"(orow));
  READ_W(0);
#pragma unroll 1
  for (int ii = 0; ii < NSTEP; ii += 2) {
    STEP(lo0, hi0, st0, me0, ii, 0);
    STEP(lo1, hi1, st1, me1, ii + 1, 1);
  }

  const float sc = anybad ? __builtin_nanf("") : scale;
  const v4f zero4 = {0.f, 0.f, 0.f, 0.f};
#pragma unroll
  for (int mf = 0; mf < 2; ++mf) {
    const v4f v0 = __builtin_elementwise_maximum(acc[0][mf] * sc, zero4);
    const v4f v1 = __builtin_elementwise_maximum(acc[1][mf] * sc, zero4);
    const v4f v2 = __builtin_elementwise_maximum(acc[2][mf] * sc, zero4);
    const v4f v3 = __builtin_elementwise_maximum(acc[3][mf] * sc, zero4);
    asm volatile("global_store_dwordx4 %0, %1, off sc1\n\tglobal_store_dwordx4 %0, %2, off offset:64 sc1\n\t"
                 "global_store_dwordx4 %0, %3, off offset:128 sc1\n\tglobal_store_dwordx4 %0, %4, off offset:192 sc1\n\ts_nop 1"
                 :: "v"(orow + mf * 16 * NS), "v"(v0), "v"(v1), "v"(v2), "v"(v3) : "memory");
  }
}

}

extern "C" void kernel_launch(int rank, int nranks, ncclComm_t comm,
                              hipStream_t stream,
                              void* const* d_in, const int* in_sizes, int n_in,
                              void* d_out, int out_size,
                              const int* in_dtypes, int out_dtype,
                              void* const* d_out_all,
                              void* const* d_ws_all, size_t ws_bytes) {
  if (nranks != NR || ws_bytes < WS_NEED || n_in != 4) { fprintf(stderr, "unexpected configuration\n"); return; }
  HIP_CHECK(hipFuncSetAttribute((const void*)ag_gemm_kernel, hipFuncAttributeMaxDynamicSharedMemorySize, LDS_BYTES));
  Params p;
  p.x = (const unsigned char*)d_in[0];
  p.w = (const unsigned char*)d_in[1];
  p.sx = (const float*)d_in[2];
  p.sw = (const float*)d_in[3];
  p.out = (float*)d_out;
  for (int i = 0; i < NR; ++i) p.ws[i] = (unsigned char*)d_ws_all[i];
  p.rank = rank;
  ag_gemm_kernel<<<NWG, NTHR, LDS_BYTES, stream>>>(p);
  HIP_CHECK(hipGetLastError());
}
```

```cpp
#include <hip/hip_runtime.h>
#include <rccl/rccl.h>
#include <cstdio>
#include <cstdint>

#define HIP_CHECK(expr) do { hipError_t _e = (expr); if (_e != hipSuccess) { \
    fprintf(stderr, "HIP error %s at %s:%d\n", hipGetErrorString(_e), __FILE__, __LINE__); \
    return; } } while (0)

namespace {

typedef int v2i __attribute__((ext_vector_type(2)));
typedef int v4i __attribute__((ext_vector_type(4)));
typedef int v8i __attribute__((ext_vector_type(8)));
typedef float v4f __attribute__((ext_vector_type(4)));
typedef unsigned u32x4 __attribute__((ext_vector_type(4)));
#define LAS __attribute__((address_space(3)))

#define IM 1
#define RAWL 2
#define PD 2
#define RAWK 1

constexpr int NR = 8;
constexpr int MS = 512;
constexpr int KD = 4096;
constexpr int NS = 512;
constexpr int BM = 64, BN = 128, BK = 128, NSTEP = KD / BK;
constexpr int A_BYTES = BM * BK, B_BYTES = BK * BN;
constexpr int LDS_B = 0, LDS_A = 4 * B_BYTES, LDS_MISC = LDS_A + 2 * A_BYTES;
constexpr int LDS_BYTES = LDS_MISC + 2048;
constexpr int NWG = 256;
constexpr int NTHR = 512;
constexpr unsigned SEC_RAW = 2048;
constexpr unsigned SEC_CMP = 1824;
constexpr unsigned BLK_CMP = 7296;
constexpr unsigned SEC_C6 = 1760;
constexpr unsigned BLK_C6 = 7040;
constexpr size_t SHARD_BYTES = (size_t)MS * KD;
constexpr size_t EXPR_OFF = 0;
constexpr size_t EXPC_OFF = 2 * SHARD_BYTES;
constexpr size_t EXP6_OFF = 4 * SHARD_BYTES;
constexpr size_t FLAG_OFF = 6 * SHARD_BYTES;
constexpr size_t SEQ_OFF = FLAG_OFF + 2 * NR * 4096;
constexpr size_t TMO_OFF = SEQ_OFF + 8192;
constexpr size_t WS_NEED = TMO_OFF + 256;
constexpr unsigned long long DEADLINE_TICKS = 100000000ull;

struct Params {
  const unsigned char* x;
  const unsigned char* w;
  const float* sx;
  const float* sw;
  float* out;
  unsigned char* ws[NR];
  int rank;
};

__device__ __forceinline__ void glds16(const void* g, unsigned lds_addr) {
  unsigned keep;
  asm volatile("s_mov_b32 %0, m0\n\ts_mov_b32 m0, %2\n\ts_nop 0\n\tglobal_load_lds_dwordx4 %1, off\n\ts_mov_b32 m0, %0"
               : "=&s"(keep) : "v"(g), "s"(lds_addr) : "memory");
}
__device__ __forceinline__ void st16x2_sys(void* p, u32x4 v0, u32x4 v1) {
  asm volatile("global_store_dwordx4 %0, %1, off sc0 sc1\n\tglobal_store_dwordx4 %0, %2, off offset:1024 sc0 sc1\n\ts_nop 1"
               :: "v"(p), "v"(v0), "v"(v1) : "memory");
}
#define WAIT_VM(n) asm volatile("s_waitcnt vmcnt(%0)" :: "i"(n) : "memory")
#define WAIT_LGKM() asm volatile("s_waitcnt lgkmcnt(0)" ::: "memory")
#define TS(i) do {} while (0)
#define TS_DUMP() do {} while (0)

__device__ __forceinline__ unsigned wave_scan(unsigned x) {
  x += __builtin_amdgcn_update_dpp(0u, x, 0x111, 0xf, 0xf, true);
  x += __builtin_amdgcn_update_dpp(0u, x, 0x112, 0xf, 0xf, true);
  x += __builtin_amdgcn_update_dpp(0u, x, 0x114, 0xf, 0xf, true);
  x += __builtin_amdgcn_update_dpp(0u, x, 0x118, 0xf, 0xf, true);
  x += __builtin_amdgcn_update_dpp(0u, x, 0x142, 0xa, 0xf, false);
  x += __builtin_amdgcn_update_dpp(0u, x, 0x143, 0xc, 0xf, false);
  return x;
}
__device__ __forceinline__ unsigned shift_pos(unsigned v, int from, int to) { return to >= from ? v << (to - from) : v >> (from - to); }
__device__ constexpr int CPOS[7] = {0, 1, 2, 3, 4, 5, 7};
__device__ constexpr int CPOS6[6] = {0, 1, 2, 3, 4, 7};
__device__ __forceinline__ bool tags_ok(u32x4 f, unsigned tag) { return (((f.x ^ tag) | (f.y ^ tag) | (f.z ^ tag) | (f.w ^ tag)) >> 8) == 0; }

__global__ __launch_bounds__(NTHR, 1) void ag_gemm_kernel(Params p) {
  extern __shared__ __attribute__((aligned(16))) unsigned char smem[];
  const int tid = threadIdx.x;
  const int lane = tid & 63;
  const int wv = __builtin_amdgcn_readfirstlane(tid >> 6);
  const int w = wv & 3;
  const int b = blockIdx.x;
  const int grp = b & 7, idx = b >> 3, rb = idx >> 2, cb = idx & 3;
  const int rank = p.rank;
  const unsigned src = (unsigned)(rank + grp) & 7u;
  unsigned char* myws = p.ws[rank];
  const unsigned lds0 = (unsigned)(uintptr_t)(LAS unsigned char*)smem;
  LAS unsigned* misc = (LAS unsigned*)(smem + LDS_MISC);
  unsigned* seqp = (unsigned*)(myws + SEQ_OFF) + b;
  unsigned* tmo = (unsigned*)(myws + TMO_OFF);
  TS(0);

  if (wv >= 4) {
    unsigned seq;
    float sxv, swv;
    asm volatile("global_load_dword %0, %3, off sc1\n\tglobal_load_dword %1, %4, off\n\tglobal_load_dword %2, %5, off"
                 : "=&v"(seq), "=&v"(sxv), "=&v"(swv) : "v"(seqp), "v"(p.sx), "v"(p.sw) : "memory");
    const unsigned char* bptr[4];
#pragma unroll
    for (int i = 0; i < 4; ++i) {
      const int kr = 32 * w + 8 * i + (lane >> 3);
      const int fb = ((kr >> 1) & 3) | (((kr >> 4) & 1) << 2);
      bptr[i] = p.w + (size_t)kr * NS + cb * BN + (((lane & 7) ^ fb) << 4);
    }
#define STAGE_B(POS) do {                                                                                        \
      const unsigned base_ = lds0 + LDS_B + (unsigned)((POS) & 3) * B_BYTES + (unsigned)w * 4096;                \
      const size_t go_ = (size_t)(POS) * (BK * NS);                                                              \
      _Pragma("unroll") for (int i = 0; i < 4; ++i) glds16(bptr[i] + go_, base_ + i * 1024);                     \
    } while (0)
    STAGE_B(0); STAGE_B(1); STAGE_B(2); STAGE_B(3);
    if (w == 0 && lane < 4) misc[lane] = 0u;
    if (w == 1 && lane < 16) {
      unsigned sd = 0, sc = 0x0C0C0C0Cu, r = 0;
#pragma unroll
      for (int j = 0; j < 4; ++j) {
        if ((lane >> j) & 1) {
          sd |= (4u + r) << (8 * j);
          sc = (sc & ~(0xFFu << (8 * r))) | ((unsigned)j << (8 * r));
          ++r;
        }
      }
      misc[208 + lane] = sd;
      misc[224 + lane] = sc;
    }
    asm volatile("s_waitcnt vmcnt(16) lgkmcnt(0)" : "+v"(seq), "+v"(sxv), "+v"(swv) :: "memory");
    __builtin_amdgcn_s_barrier();
    seq = __builtin_amdgcn_readfirstlane(seq) + 1u;
    const float scale = sxv * swv;
    if (w == 0 && grp != 0) {
      const unsigned q = seq & 1u, tag = (seq | 0x800000u) << 8;
      const unsigned* fptr = (const unsigned*)(myws + FLAG_OFF) + ((q * NR + src) * 8 + rb) * 128 + (lane < RAWK ? lane : 0) * 4;
      const unsigned long long t0 = __builtin_amdgcn_s_memrealtime();
      for (unsigned it = 1;; ++it) {
        u32x4 f;
        asm volatile("global_load_dwordx4 %0, %1, off sc0 sc1\n\ts_waitcnt vmcnt(0)" : "=v"(f) : "v"(fptr) : "memory");
        if (__all(tags_ok(f, tag))) {
          const unsigned char* g = p.ws[src] + EXPR_OFF + (size_t)q * SHARD_BYTES + (size_t)(rb * 4 + cb) * SEC_RAW + lane * 16;
#pragma unroll
          for (int k = 0; k < RAWK; ++k) {
            glds16(g + (size_t)k * (8 * 8192), lds0 + LDS_MISC + 1024);
            glds16(g + (size_t)k * (8 * 8192) + 1024, lds0 + LDS_MISC + 1024);
          }
          break;
        }
        if ((it & 63u) == 0) {
          if (__hip_atomic_load(tmo, __ATOMIC_RELAXED, __HIP_MEMORY_SCOPE_AGENT) == seq ||
              __builtin_amdgcn_s_memrealtime() - t0 > DEADLINE_TICKS) break;
        }
        if (it > 64u) __builtin_amdgcn_s_sleep(8);
      }
    } else
    WAIT_VM(8);
    const int wm = w >> 1, wn = w & 1, fr = lane & 15, g = lane >> 4;
    const unsigned xo0 = (32 * wm + fr) * 128 + ((g ^ ((fr >> 1) & 7)) << 4);
    const unsigned xo1 = (32 * wm + fr) * 128 + (((g + 4) ^ ((fr >> 1) & 7)) << 4);
    unsigned wo[4];
#pragma unroll
    for (int nf = 0; nf < 4; ++nf)
      wo[nf] = (16 * g + (fr >> 1)) * 128 + (((4 * wn + nf) ^ (((fr >> 2) & 3) | ((g & 1) << 2))) << 4) + 8 * (fr & 1);
    float* orow = p.out + (size_t)(src * MS + rb * BM + 32 * wm + fr) * NS + cb * BN + 64 * wn + 4 * g;
    asm volatile("" : "+v"(orow));
    v4f acc[4][2];
#pragma unroll
    for (int nf = 0; nf < 4; ++nf)
#pragma unroll
      for (int mf = 0; mf < 2; ++mf) acc[nf][mf] = (v4f){0.f, 0.f, 0.f, 0.f};
    v8i wf[4];
#define READ_W(POS) do {                                                                                         \
      const unsigned char* sb_ = smem + LDS_B + ((POS) & 3) * B_BYTES;                                           \
      _Pragma("unroll") for (int nf = 0; nf < 4; ++nf) {                                                         \
        v2i t_[4];                                                                                               \
        _Pragma("unroll") for (int c = 0; c < 4; ++c)                                                            \
          t_[c] = __builtin_amdgcn_ds_read_tr8_b64_v2i32((LAS v2i*)(sb_ + wo[nf] + (64 * (c >> 1) + 8 * (c & 1)) * 128)); \
        wf[nf] = (v8i){t_[0].x, t_[0].y, t_[1].x, t_[1].y, t_[2].x, t_[2].y, t_[3].x, t_[3].y};                  \
      }                                                                                                          \
    } while (0)
    __builtin_amdgcn_s_barrier();
    const bool anybad = misc[4] != 0;
    READ_W(0);
    WAIT_LGKM();
    __builtin_amdgcn_s_barrier();
#pragma unroll 1
    for (int k = 1; k <= NSTEP; ++k) {
      const unsigned char* sa = smem + LDS_A + ((k - 1) & 1) * A_BYTES;
      v8i xf[2];
#pragma unroll
      for (int mf = 0; mf < 2; ++mf) {
        const v4i l_ = *(const v4i*)(sa + xo0 + mf * 2048);
        const v4i h_ = *(const v4i*)(sa + xo1 + mf * 2048);
        xf[mf] = (v8i){l_.x, l_.y, l_.z, l_.w, h_.x, h_.y, h_.z, h_.w};
      }
      if (k + 3 < NSTEP) STAGE_B(k + 3);
#pragma unroll
      for (int mf = 0; mf < 2; ++mf)
#pragma unroll
        for (int nf = 0; nf < 4; ++nf)
          acc[nf][mf] = __builtin_amdgcn_mfma_scale_f32_16x16x128_f8f6f4(wf[nf], xf[mf], acc[nf][mf], 1, 0, 0, 127, 0, 127);
      asm volatile("" : "+v"(acc[0][0]), "+v"(acc[0][1]), "+v"(acc[1][0]), "+v"(acc[1][1]),
                        "+v"(acc[2][0]), "+v"(acc[2][1]), "+v"(acc[3][0]), "+v"(acc[3][1]));
      if (k == NSTEP) break;
      READ_W(k);
      if (k <= NSTEP - 4) WAIT_VM(8);
      else if (k == NSTEP - 3) WAIT_VM(4);
      else WAIT_VM(0);
      WAIT_LGKM();
      __builtin_amdgcn_s_barrier();
    }
    TS(39);
    const float sc = anybad ? __builtin_nanf("") : scale;
    const v4f zero4 = {0.f, 0.f, 0.f, 0.f};
#pragma unroll
    for (int mf = 0; mf < 2; ++mf) {
      const v4f v0 = __builtin_elementwise_maximum(acc[0][mf] * sc, zero4);
      const v4f v1 = __builtin_elementwise_maximum(acc[1][mf] * sc, zero4);
      const v4f v2 = __builtin_elementwise_maximum(acc[2][mf] * sc, zero4);
      const v4f v3 = __builtin_elementwise_maximum(acc[3][mf] * sc, zero4);
      asm volatile("global_store_dwordx4 %0, %1, off sc1\n\tglobal_store_dwordx4 %0, %2, off offset:64 sc1\n\t"
                   "global_store_dwordx4 %0, %3, off offset:128 sc1\n\tglobal_store_dwordx4 %0, %4, off offset:192 sc1\n\ts_nop 1"
                   :: "v"(orow + mf * 16 * NS), "v"(v0), "v"(v1), "v"(v2), "v"(v3) : "memory");
    }
    return;
  }

  const unsigned rowoff = (unsigned)(16 * w + (lane >> 2)) * KD + (lane & 3) * 16;
  const unsigned char* xp = p.x + (size_t)(grp * BM) * KD + idx * BK + rowoff;
  unsigned char* peerws = p.ws[(unsigned)(rank + lane) & 7u];
  u32x4 xv0, xv1;
  unsigned seq;
  asm volatile("global_load_dwordx4 %0, %3, off\n\tglobal_load_dwordx4 %1, %3, off offset:64\n\tglobal_load_dword %2, %4, off sc1\n\t"
               "s_waitcnt vmcnt(0)"
               : "=&v"(xv0), "=&v"(xv1), "=&v"(seq) : "v"(xp), "v"(seqp), "v"(peerws) : "memory");
  __builtin_amdgcn_s_barrier();
  seq = __builtin_amdgcn_readfirstlane(seq) + 1u;
  const unsigned q = seq & 1u;
  const unsigned tag = (seq | 0x800000u) << 8;
  unsigned meta = 0;
  if (idx < RAWK || idx >= NSTEP - RAWL) {
    st16x2_sys(myws + EXPR_OFF + (size_t)q * SHARD_BYTES + (size_t)(b * 4 + w) * SEC_RAW + lane * 16, xv0, xv1);
  } else {
    unsigned D[8] = {xv0.x, xv0.y, xv0.z, xv0.w, xv1.x, xv1.y, xv1.z, xv1.w};
    LAS unsigned char* scr = (LAS unsigned char*)smem + LDS_A + w * 2048;
    constexpr int NIB = A_BYTES;
    unsigned base, b3;
    {
      unsigned sum = 0;
#pragma unroll
      for (int i = 0; i < 8; ++i) sum = __builtin_amdgcn_sad_u8(D[i] & 0x78787878u, 0u, sum);
      const int tot = (int)__builtin_amdgcn_readlane(wave_scan(sum), 63);
      const int t = (tot - 54067) >> 14, t3 = (tot - 819) >> 14;
      base = t < 0 ? 0u : (t > 8 ? 8u : (unsigned)t);
      b3 = t3 < 0 ? 0u : (t3 > 12 ? 12u : (unsigned)t3);
    }
    bool done = false;
    {
      if (lane < 60) *(LAS unsigned*)(scr + NIB + lane * 4) = 0u;
      const unsigned k1 = (128u - 8u * b3) * 0x01010101u, k3 = (104u - 8u * b3) * 0x01010101u;
      const unsigned kLo = (b3 >= 6u ? 176u - 8u * b3 : 128u) * 0x01010101u;
      const unsigned kHi = (b3 <= 10u ? 88u - 8u * b3 : 0u) * 0x01010101u;
      const unsigned kC = (64u + ((48u - 8u * b3) & 63u)) * 0x01010101u;
      unsigned C[8], c2[8], sel[8], run[9], far = 0;
      run[0] = 0;
#pragma unroll
      for (int i = 0; i < 8; ++i) {
        const unsigned u = D[i] & 0x78787878u;
        const unsigned g1 = u + k1, g3 = u + k3;
        const unsigned esc = (~g1 | g3) & 0x80808080u, h = g3 & 0x80808080u;
        far |= ~(u + kLo) | (u + kHi);
        C[i] = (D[i] & 0x87878787u) | (g1 & 0x18181818u) | ((esc - (esc >> 2)) >> 2);
        c2[i] = ((u + kC - ((h - (h >> 2)) >> 2)) >> 3) & 0x07070707u;
        sel[i] = misc[224 + (__builtin_amdgcn_udot4(esc, 0x08040201u, 0u, false) >> 7)];
        run[i + 1] = run[i] + __builtin_popcount(esc);
      }
      const unsigned cnt = run[8];
      const unsigned inc = wave_scan(cnt);
      const unsigned total = __builtin_amdgcn_readlane(inc, 63);
      if (total <= 597u && !__any((far & 0x80808080u) != 0u || cnt > 21u)) {
        unsigned long long acc = 0;
#pragma unroll
        for (int i = 0; i < 8; ++i) {
          const unsigned y = __builtin_amdgcn_perm(0u, c2[i], sel[i]);
          const unsigned v = __builtin_amdgcn_udot4(y, 0x00400801u, 0u, false) + ((y >> 24) << 9);
          acc |= (unsigned long long)v << (3u * run[i]);
        }
#pragma unroll
        for (int i = 0; i < 6; ++i)
          C[i] |= (shift_pos(C[6], CPOS6[i], 6) & 0x40404040u) | (shift_pos(C[7], CPOS6[i], 5) & 0x20202020u);
        const unsigned bp = (inc - cnt) * 3u, sh = bp & 31u;
        LAS unsigned* sp = (LAS unsigned*)(scr + NIB) + (bp >> 5);
        const unsigned long long a0 = acc << sh;
        __hip_atomic_fetch_or(sp, (unsigned)a0, __ATOMIC_RELAXED, __HIP_MEMORY_SCOPE_WORKGROUP);
        __hip_atomic_fetch_or(sp + 1, (unsigned)(a0 >> 32), __ATOMIC_RELAXED, __HIP_MEMORY_SCOPE_WORKGROUP);
        __hip_atomic_fetch_or(sp + 2, (unsigned)((acc >> 32) << sh >> 32), __ATOMIC_RELAXED, __HIP_MEMORY_SCOPE_WORKGROUP);
        const unsigned stv = *(volatile LAS unsigned*)(scr + NIB + (lane < 56 ? lane : 55) * 4);
        unsigned char* sec = myws + EXP6_OFF + (size_t)q * SHARD_BYTES + (size_t)b * BLK_C6 + w * SEC_C6;
        typedef unsigned u32x2 __attribute__((ext_vector_type(2)));
        const u32x4 c0 = {C[0], C[1], C[2], C[3]};
        const u32x2 c1 = {C[4], C[5]};
        asm volatile("global_store_dwordx4 %0, %3, off sc0 sc1\n\tglobal_store_dwordx2 %1, %4, off sc0 sc1\n\t"
                     "global_store_dword %2, %5, off sc0 sc1\n\ts_nop 1"
                     :: "v"(sec + lane * 16), "v"(sec + 1024 + lane * 8), "v"(sec + 1536 + (lane < 56 ? lane : 55) * 4),
                        "v"(c0), "v"(c1), "v"(stv) : "memory");
        meta = 0x40u | b3;
        done = true;
      }
    }
    if (!done) {
    *(LAS u32x4*)(scr + lane * 32) = xv0;
    *(LAS u32x4*)(scr + lane * 32 + 16) = xv1;
    if (lane < 16) *(LAS unsigned*)(scr + NIB + lane * 4) = 0u;
    const unsigned cB = (128u - 8u * base) * 0x01010101u, cB7 = (72u - 8u * base) * 0x01010101u;
    const unsigned cLo = (base >= 6u ? 176u - 8u * base : 128u) * 0x01010101u;
    const unsigned cHi = (base <= 6u ? 56u - 8u * base : 0u) * 0x01010101u;
    unsigned C[8], M = 0, far = 0;
#pragma unroll
    for (int i = 0; i < 8; ++i) {
      const unsigned u = D[i] & 0x78787878u;
      const unsigned g1 = u + cB, g2 = u + cB7;
      const unsigned esc = (~g1 | g2) & 0x80808080u;
      far |= ~(u + cLo) | (u + cHi);
      C[i] = (D[i] & 0x87878787u) | (g1 & 0x38383838u) | ((esc - (esc >> 3)) >> 1);
      M |= esc >> (7 - i);
    }
#pragma unroll
    for (int i = 0; i < 7; ++i) C[i] |= shift_pos(C[7], CPOS[i], 6) & 0x40404040u;
    const unsigned cnt = __builtin_popcount(M);
    const unsigned inc = wave_scan(cnt);
    const unsigned total = __builtin_amdgcn_readlane(inc, 63);
    const bool use3 = total <= 85u && !__any((far & 0x80808080u) != 0u);
    if ((use3 || total <= 64u) && !__any(cnt > 8u)) {
      const unsigned wd = use3 ? 3u : 4u;
      unsigned S = 0, Mm = M;
      unsigned by[8];
#pragma unroll
      for (int k = 0; k < 8; ++k) {
        const unsigned bt = (unsigned)__builtin_ctz(Mm | 0x80000000u);
        by[k] = *(LAS unsigned char*)(scr + lane * 32 + ((bt & 7) << 2) + (bt >> 3));
        Mm &= Mm - 1;
      }
#pragma unroll
      for (int k = 0; k < 8; ++k) {
        const unsigned e = (by[k] >> 3) & 15u;
        const int d = (int)e - (int)base;
        const unsigned code = use3 ? (unsigned)(d < 0 ? -1 - d : d - 1) : e;
        S |= (k < (int)cnt ? code : 0u) << (wd * k);
      }
      const unsigned bp = (inc - cnt) * wd, sh = bp & 31u;
      if (cnt) {
        LAS unsigned* sp = (LAS unsigned*)(scr + NIB) + (bp >> 5);
        __hip_atomic_fetch_or(sp, S << sh, __ATOMIC_RELAXED, __HIP_MEMORY_SCOPE_WORKGROUP);
        __hip_atomic_fetch_or(sp + 1, (unsigned)(((unsigned long long)S << sh) >> 32), __ATOMIC_RELAXED, __HIP_MEMORY_SCOPE_WORKGROUP);
      }
      const unsigned stv = *(volatile LAS unsigned*)(scr + NIB + (lane < 8 ? lane : 7) * 4);
      unsigned char* sec = myws + EXPC_OFF + (size_t)q * SHARD_BYTES + (size_t)b * BLK_CMP + w * SEC_CMP;
      typedef unsigned u32x3 __attribute__((ext_vector_type(3)));
      const u32x4 c0 = {C[0], C[1], C[2], C[3]};
      const u32x3 c1 = {C[4], C[5], C[6]};
      asm volatile("global_store_dwordx4 %0, %3, off sc0 sc1\n\tglobal_store_dwordx3 %1, %4, off sc0 sc1\n\t"
                   "global_store_dword %2, %5, off sc0 sc1\n\ts_nop 1"
                   :: "v"(sec + lane * 16), "v"(sec + 1024 + lane * 12), "v"(sec + 1792 + (lane < 8 ? lane : 7) * 4),
                      "v"(c0), "v"(c1), "v"(stv) : "memory");
      meta = (use3 ? 0xC0u : 0x80u) | base;
    } else {
      st16x2_sys(myws + EXPR_OFF + (size_t)q * SHARD_BYTES + (size_t)(b * 4 + w) * SEC_RAW + lane * 16, xv0, xv1);
    }
    }
  }
  TS(1);
  WAIT_VM(0);
  if (lane == 0) *(volatile LAS unsigned*)(misc + w) = tag | meta;
  TS(2);
  if (w == 3) {
    u32x4 fv;
    do {
      asm volatile("ds_read_b128 %0, %1\n\ts_waitcnt lgkmcnt(0)" : "=v"(fv) : "v"(lds0 + LDS_MISC) : "memory");
    } while (fv.x == 0u || fv.y == 0u || fv.z == 0u || fv.w == 0u);
    if (lane >= 1 && lane < NR)
      asm volatile("global_store_dwordx4 %0, %1, off sc0 sc1\n\ts_nop 1"
                   :: "v"((unsigned*)(peerws + FLAG_OFF) + ((q * NR + rank) * 8 + grp) * 128 + idx * 4), "v"(fv) : "memory");
    if (lane == 0) __hip_atomic_store(seqp, seq, __ATOMIC_RELAXED, __HIP_MEMORY_SCOPE_AGENT);
  } else if (w == 0) {
    unsigned bad = 0;
    u32x4 f = {0u, 0u, 0u, 0u};
    if (grp != 0) {
      const unsigned* fptr = (const unsigned*)(myws + FLAG_OFF) + ((q * NR + src) * 8 + rb) * 128 + (lane & 31) * 4;
      const unsigned long long t0 = __builtin_amdgcn_s_memrealtime();
      for (unsigned it = 1;; ++it) {
        asm volatile("global_load_dwordx4 %0, %1, off sc0 sc1\n\ts_waitcnt vmcnt(0)" : "=v"(f) : "v"(fptr) : "memory");
        if (__all(tags_ok(f, tag))) break;
        if (it > 64u) __builtin_amdgcn_s_sleep(8);
        if ((it & 63u) == 0 && (__hip_atomic_load(tmo, __ATOMIC_RELAXED, __HIP_MEMORY_SCOPE_AGENT) == seq ||
                                __builtin_amdgcn_s_memrealtime() - t0 > DEADLINE_TICKS)) {
          if (lane == 0) __hip_atomic_store(tmo, seq, __ATOMIC_RELAXED, __HIP_MEMORY_SCOPE_AGENT);
          bad = 1;
          f = (u32x4){0u, 0u, 0u, 0u};
          break;
        }
      }
    }
    if (lane < 32) *(LAS u32x4*)(misc + 64 + lane * 4) = f;
    if (lane == 0) misc[4] = bad;
    WAIT_LGKM();
  }
  __builtin_amdgcn_s_barrier();
  const unsigned flags = misc[64 + (lane & 31) * 4 + w];
  TS(3);

  const unsigned o1 = grp == 0 ? rowoff : (unsigned)lane * 16;
  const unsigned o2 = grp == 0 ? rowoff + 64 : 1024u + lane * 16;
  const unsigned char* raw_base = grp == 0 ? p.x + (size_t)(rb * BM) * KD
                                           : p.ws[src] + EXPR_OFF + (size_t)q * SHARD_BYTES + (size_t)(rb * 4 + w) * SEC_RAW;
  const unsigned raw_step = grp == 0 ? BK : 8 * 8192;
  const unsigned char* cmp_base = p.ws[src] + EXPC_OFF + (size_t)q * SHARD_BYTES + (size_t)rb * BLK_CMP + w * SEC_CMP;
  const unsigned char* c6_base = p.ws[src] + EXP6_OFF + (size_t)q * SHARD_BYTES + (size_t)rb * BLK_C6 + w * SEC_C6;
  const unsigned o2c = 1024u + lane * 12, o3c = 1792u + (lane < 8 ? lane : 7) * 4;
  const unsigned o2s = 1024u + lane * 8, o3s = 1536u + (lane < 56 ? lane : 55) * 4;
#define LOAD_A(LO, HI, ST, ME, POS) do {                                                                         \
    const int ks_ = (POS);                                                                                       \
    ME = __builtin_amdgcn_readlane(flags, ks_) & 0xFFu;                                                          \
    const bool c_ = (ME & 0x80u) != 0, s_ = (ME & 0xC0u) == 0x40u;                                               \
    const unsigned char* base_ = (s_ ? c6_base : (c_ ? cmp_base : raw_base)) +                                   \
                                 (unsigned)ks_ * (s_ ? 8u * BLK_C6 : (c_ ? 8u * BLK_CMP : raw_step));            \
    const unsigned oa_ = s_ ? o2s : (c_ ? o2c : o2), ob_ = s_ ? o3s : (c_ ? o3c : o1);                           \
    asm volatile("global_load_dwordx4 %0, %3, %6\n\tglobal_load_dwordx4 %1, %4, %6\n\tglobal_load_dword %2, %5, %6" \
                 : "=&v"(LO), "=&v"(HI), "=&v"(ST) : "v"(o1), "v"(oa_), "v"(ob_), "s"(base_) : "memory");        \
  } while (0)
  static_assert(PD == 2 && IM >= 1, "two register sets: a set is reloaded once it is unpacked");
  u32x4 lo0, hi0, lo1, hi1;
  unsigned st0, st1, me0, me1;
  LOAD_A(lo0, hi0, st0, me0, 0);
  LOAD_A(lo1, hi1, st1, me1, 1);
  unsigned awr0, awr1;
  {
    const int r = 16 * w + (lane >> 2), fa = (r >> 1) & 7;
    awr0 = r * 128 + (((lane & 3) ^ fa) << 4);
    awr1 = r * 128 + ((((lane & 3) + 4) ^ fa) << 4);
  }
#define STEP(LO, HI, ST, ME, NLO, NHI, NST, NME, I, U) do {                                                                          \
    const int i = (I);                                                                                           \
    if (i == 16) TS(44);                                                                                         \
    if (i <= NSTEP - PD) WAIT_VM(3 * (PD - 1)); else if (i == NSTEP - 2) WAIT_VM(3); else WAIT_VM(0);            \
    asm volatile("" : "+v"(LO), "+v"(HI), "+v"(ST));                                                             \
    TS(4 + i);                                                                                                   \
    if (IM == 0 && i + PD < NSTEP) LOAD_A(NLO, NHI, NST, NME, i + PD);                                           \
    unsigned char* sa = smem + LDS_A + (U) * A_BYTES;                                                            \
    if ((ME & 0xC0u) == 0x40u) {                                                                                 \
      const unsigned b3 = (ME & 15u) * 0x01010101u;                                                              \
                          \
      const unsigned LL = (((b3 + 0x0D0C0B0Au) & 0x0F0F0F0Fu) ^ (b3 + 0x03030303u)) << 3;                        \
      const unsigned LH = (((b3 + 0x14130F0Eu) & 0x0F0F0F0Fu) ^ (b3 + 0x03030303u)) << 3;                        \
      unsigned C[8] = {LO.x, LO.y, LO.z, LO.w, HI.x, HI.y, 0u, 0u};                                              \
      _Pragma("unroll") for (int k = 0; k < 6; ++k) {                                                            \
        C[6] |= shift_pos(C[k] & 0x40404040u, 6, CPOS6[k]);                                                      \
        C[7] |= shift_pos(C[k] & 0x20202020u, 5, CPOS6[k]);                                                      \
      }                                                                                                          \
      unsigned sel[8], run[9];                                                                                   \
      run[0] = 0;                                                                                                \
      _Pragma("unroll") for (int k = 0; k < 8; ++k) {                                                            \
        C[k] &= 0x9F9F9F9Fu;                                                                                     \
        const unsigned f = (C[k] + 0x08080808u) & 0x20202020u;                                                   \
        sel[k] = misc[208 + (__builtin_amdgcn_udot4(f, 0x08040201u, 0u, false) >> 5)];                           \
        run[k + 1] = run[k] + __builtin_popcount(f);                                                             \
      }                                                                                                          \
      const unsigned bp = (wave_scan(run[8]) - run[8]) * 3u;                                                     \
      const unsigned w0 = __builtin_amdgcn_ds_bpermute((int)((bp >> 5) << 2), (int)ST);                          \
      const unsigned w1 = __builtin_amdgcn_ds_bpermute((int)(((bp >> 5) + 1) << 2), (int)ST);                    \
      const unsigned w2 = __builtin_amdgcn_ds_bpermute((int)(((bp >> 5) + 2) << 2), (int)ST);                    \
      const unsigned long long W = ((unsigned long long)__builtin_amdgcn_alignbit(w2, w1, bp & 31u) << 32) |     \
                                   __builtin_amdgcn_alignbit(w1, w0, bp & 31u);                                  \
      _Pragma("unroll") for (int k = 0; k < 8; ++k) {                                                            \
        const unsigned v = (unsigned)(W >> (3u * run[k]));                                                       \
        const unsigned y = (((v & 0x1C7u) * 0x401u) & 0x00070007u) |                  \
                           (((v & 0xE38u) * 0x8020u) & 0x07000700u);                  \
        const unsigned z = __builtin_amdgcn_perm(y, 0x0C0C0C0Cu, sel[k]);                                        \
        C[k] = (C[k] + (b3 << 3)) ^ __builtin_amdgcn_perm(LH, LL, z);                                            \
      }                                                                                                          \
      *(u32x4*)(sa + awr0) = (u32x4){C[0], C[1], C[2], C[3]};                                                    \
      *(u32x4*)(sa + awr1) = (u32x4){C[4], C[5], C[6], C[7]};                                                    \
    } else if (ME & 0x80u) {                                                                                     \
      const unsigned base = ME & 15u;                                                                            \
      unsigned C[8] = {LO.x, LO.y, LO.z, LO.w, HI.x, HI.y, HI.z, 0u};                                            \
      _Pragma("unroll") for (int k = 0; k < 7; ++k) C[7] |= shift_pos(C[k] & 0x40404040u, 6, CPOS[k]);           \
      unsigned M = 0, Dd[8];                                                                                     \
      _Pragma("unroll") for (int k = 0; k < 8; ++k) {                                                            \
        const unsigned t = C[k] & 0xBFBFBFBFu;                                                                   \
        M |= shift_pos((t + 0x08080808u) & 0x40404040u, 6, k);                                                   \
        Dd[k] = t + base * 0x08080808u;                                                                          \
      }                                                                                                          \
      *(u32x4*)(sa + awr0) = (u32x4){Dd[0], Dd[1], Dd[2], Dd[3]};                                                \
      *(u32x4*)(sa + awr1) = (u32x4){Dd[4], Dd[5], Dd[6], Dd[7]};                                                \
      const unsigned cnt = __builtin_popcount(M);                                                                \
      const bool use3 = (ME & 0x40u) != 0;                                                                       \
      const unsigned wd = use3 ? 3u : 4u;                                                                        \
      const unsigned bp = (wave_scan(cnt) - cnt) * wd;                                                           \
      const unsigned w0 = __builtin_amdgcn_ds_bpermute((int)((bp >> 5) << 2), (int)ST);                          \
      const unsigned w1 = __builtin_amdgcn_ds_bpermute((int)(((bp >> 5) + 1) << 2), (int)ST);                    \
      unsigned S = __builtin_amdgcn_alignbit(w1, w0, bp & 31u);                                                  \
                          \
      const unsigned abase = (unsigned)(uintptr_t)(LAS unsigned char*)sa + awr0;                                 \
      if (use3) {                                                                                                \
        while (__any(M != 0u)) {                                                                                 \
          const unsigned bt = (unsigned)__builtin_ctz(M | 0x80000000u);                                          \
          const unsigned c = S & 7u;                                                                             \
          unsigned dl = c < 6u ? 0u - 8u - c : c - 6u;                                                           \
          dl = M ? dl : 0u;                                                                                      \
          M &= M - 1;                                                                                            \
          S >>= 3;                                                                                               \
          asm volatile("ds_add_u32 %0, %1" :: "v"(abase ^ (((bt << 4) & 64u) | ((bt << 2) & 12u))),              \
                       "v"(dl << ((bt & 24u) | 3u)) : "memory");                                                 \
        }                                                                                                        \
      } else {                                                                                                   \
        while (__any(M != 0u)) {                                                                                 \
          const unsigned bt = (unsigned)__builtin_ctz(M | 0x80000000u);                                          \
          const unsigned dl = M ? (S & 15u) - base - 7u : 0u;                                                    \
          M &= M - 1;                                                                                            \
          S >>= 4;                                                                                               \
          asm volatile("ds_add_u32 %0, %1" :: "v"(abase ^ (((bt << 4) & 64u) | ((bt << 2) & 12u))),              \
                       "v"(dl << ((bt & 24u) | 3u)) : "memory");                                                 \
        }                                                                                                        \
      }                                                                                                          \
    } else {                                                                                                     \
      *(u32x4*)(sa + awr0) = LO;                                                                                 \
      *(u32x4*)(sa + awr1) = HI;                                                                                 \
    }                                                                                                            \
    WAIT_LGKM();                                                               \
    if (IM == 1 && i + PD < NSTEP) LOAD_A(NLO, NHI, NST, NME, i + PD);                                           \
    if (i == 16) TS(45);                                                                                         \
    __builtin_amdgcn_s_barrier();                                                                         \
    if (i == 16) TS(46);                                                                                         \
    if (IM == 2 && i + PD < NSTEP) LOAD_A(NLO, NHI, NST, NME, i + PD);                                           \
  } while (0)
#pragma unroll 1
  for (int ii = 0; ii < NSTEP; ii += 2) {
    STEP(lo0, hi0, st0, me0, lo0, hi0, st0, me0, ii, 0);
    STEP(lo1, hi1, st1, me1, lo1, hi1, st1, me1, ii + 1, 1);
  }
  TS_DUMP();
}

}

extern "C" void kernel_launch(int rank, int nranks, ncclComm_t comm,
                              hipStream_t stream,
                              void* const* d_in, const int* in_sizes, int n_in,
                              void* d_out, int out_size,
                              const int* in_dtypes, int out_dtype,
                              void* const* d_out_all,
                              void* const* d_ws_all, size_t ws_bytes) {
  if (nranks != NR || ws_bytes < WS_NEED || n_in != 4) { fprintf(stderr, "unexpected configuration\n"); return; }
  HIP_CHECK(hipFuncSetAttribute((const void*)ag_gemm_kernel, hipFuncAttributeMaxDynamicSharedMemorySize, LDS_BYTES));
  Params p;
  p.x = (const unsigned char*)d_in[0];
  p.w = (const unsigned char*)d_in[1];
  p.sx = (const float*)d_in[2];
  p.sw = (const float*)d_in[3];
  p.out = (float*)d_out;
  for (int i = 0; i < NR; ++i) p.ws[i] = (unsigned char*)d_ws_all[i];
  p.rank = rank;
  ag_gemm_kernel<<<NWG, NTHR, LDS_BYTES, stream>>>(p);
  HIP_CHECK(hipGetLastError());
}
```

```cpp
#include <hip/hip_runtime.h>
#include <rccl/rccl.h>
#include <cstdio>
#include <cstdint>
#include <cstddef>

#define HIP_CHECK(expr) do { hipError_t _e = (expr); if (_e != hipSuccess) { \
    fprintf(stderr, "HIP error %s at %s:%d\n", hipGetErrorString(_e), __FILE__, __LINE__); \
    return; } } while (0)

namespace {

typedef int v2i __attribute__((ext_vector_type(2)));
typedef int v4i __attribute__((ext_vector_type(4)));
typedef int v8i __attribute__((ext_vector_type(8)));
typedef float v4f __attribute__((ext_vector_type(4)));
typedef unsigned u32x4 __attribute__((ext_vector_type(4)));
#define LAS __attribute__((address_space(3)))

#define RAWK 1
#define RAWL 2

constexpr int NR = 8;
constexpr int MS = 512;
constexpr int KD = 4096;
constexpr int NS = 512;
constexpr int BM = 64, BN = 128, BK = 128, NSTEP = KD / BK;
constexpr int A_BYTES = BM * BK, B_BYTES = BK * BN;
constexpr int LDS_B = 0, LDS_A = 4 * B_BYTES, LDS_MISC = LDS_A + 2 * A_BYTES;
constexpr int LDS_BYTES = LDS_MISC + 2304;
constexpr int NWG = 256;
constexpr int NTHR = 512;
constexpr unsigned SEC_RAW = 2048;
constexpr unsigned SEC_CMP = 1824;
constexpr unsigned BLK_CMP = 7296;
constexpr unsigned SEC_C6 = 1760;
constexpr unsigned BLK_C6 = 7040;
constexpr size_t SHARD_BYTES = (size_t)MS * KD;
constexpr size_t EXPR_OFF = 0;
constexpr size_t EXPC_OFF = 2 * SHARD_BYTES;
constexpr size_t EXP6_OFF = 4 * SHARD_BYTES;
constexpr size_t FLAG_OFF = 6 * SHARD_BYTES;
constexpr size_t SEQ_OFF = FLAG_OFF + 2 * NR * 4096;
constexpr size_t TMO_OFF = SEQ_OFF + 8192;
constexpr size_t WS_NEED = TMO_OFF + 256;
constexpr unsigned long long DEADLINE_TICKS = 100000000ull;

struct Params {
  const unsigned char* x;
  const unsigned char* w;
  const float* sx;
  const float* sw;
  float* out;
  unsigned char* myws;
  unsigned char* ws[NR];
  int rank;
};

__device__ __forceinline__ void glds16(const void* g, unsigned lds_addr) {
  unsigned keep;
  asm volatile("s_mov_b32 %0, m0\n\ts_mov_b32 m0, %2\n\ts_nop 0\n\tglobal_load_lds_dwordx4 %1, off\n\ts_mov_b32 m0, %0"
               : "=&s"(keep) : "v"(g), "s"(lds_addr) : "memory");
}
__device__ __forceinline__ void st16x2_sys(void* p, u32x4 v0, u32x4 v1) {
  asm volatile("global_store_dwordx4 %0, %1, off sc0 sc1\n\tglobal_store_dwordx4 %0, %2, off offset:1024 sc0 sc1\n\ts_nop 1"
               :: "v"(p), "v"(v0), "v"(v1) : "memory");
}
#define WAIT_VM(n) asm volatile("s_waitcnt vmcnt(%0)" :: "i"(n) : "memory")
#define WAIT_LGKM() asm volatile("s_waitcnt lgkmcnt(0)" ::: "memory")
#define TS(i) do {} while (0)
#define TS_DUMP() do {} while (0)

__device__ __forceinline__ unsigned wave_scan(unsigned x) {
  x += __builtin_amdgcn_update_dpp(0u, x, 0x111, 0xf, 0xf, true);
  x += __builtin_amdgcn_update_dpp(0u, x, 0x112, 0xf, 0xf, true);
  x += __builtin_amdgcn_update_dpp(0u, x, 0x114, 0xf, 0xf, true);
  x += __builtin_amdgcn_update_dpp(0u, x, 0x118, 0xf, 0xf, true);
  x += __builtin_amdgcn_update_dpp(0u, x, 0x142, 0xa, 0xf, false);
  x += __builtin_amdgcn_update_dpp(0u, x, 0x143, 0xc, 0xf, false);
  return x;
}
__device__ __forceinline__ unsigned shift_pos(unsigned v, int from, int to) { return to >= from ? v << (to - from) : v >> (from - to); }
__device__ constexpr int CPOS[7] = {0, 1, 2, 3, 4, 5, 7};
__device__ __forceinline__ unsigned lds_u32(unsigned addr) { return *(LAS const unsigned*)(uintptr_t)addr; }
__device__ __forceinline__ bool tags_ok(u32x4 f, unsigned tag) { return (((f.x ^ tag) | (f.y ^ tag) | (f.z ^ tag) | (f.w ^ tag)) >> 8) == 0; }

__global__ __launch_bounds__(NTHR, 1) void ag_gemm_kernel(Params p) {
  extern __shared__ __attribute__((aligned(16))) unsigned char smem[];
  const int tid = threadIdx.x;
  const int lane = tid & 63;
  const int wv = __builtin_amdgcn_readfirstlane(tid >> 6);
  const int w = wv & 3;
  const int b = blockIdx.x;
  const int grp = b & 7, idx = b >> 3, rb = idx >> 2, cb = idx & 3;
  const int rank = p.rank;
  const unsigned src = (unsigned)(rank + grp) & 7u;
  unsigned char* myws = p.myws;
  asm volatile("" :: "s"(p.x), "s"(p.w), "s"(p.sx), "s"(p.sw), "s"(myws), "s"(rank));
  const unsigned lds0 = (unsigned)(uintptr_t)(LAS unsigned char*)smem;
  LAS unsigned* misc = (LAS unsigned*)(smem + LDS_MISC);
  unsigned* seqp = (unsigned*)(myws + SEQ_OFF) + b;
  unsigned* tmo = (unsigned*)(myws + TMO_OFF);
  TS(0);

  if (wv >= 4) {
    unsigned seq;
    float sxv, swv;
    asm volatile("global_load_dword %0, %3, off sc1\n\tglobal_load_dword %1, %4, off\n\tglobal_load_dword %2, %5, off"
                 : "=&v"(seq), "=&v"(sxv), "=&v"(swv) : "v"(seqp), "v"(p.sx), "v"(p.sw) : "memory");
    if (w == 0 && lane < 4) misc[lane] = 0u;
    WAIT_LGKM();
    __builtin_amdgcn_s_barrier();
    const unsigned char* bptr[4];
#pragma unroll
    for (int i = 0; i < 4; ++i) {
      const int kr = 32 * w + 8 * i + (lane >> 3);
      const int fb = ((kr >> 1) & 3) | (((kr >> 4) & 1) << 2);
      bptr[i] = p.w + (size_t)kr * NS + cb * BN + (((lane & 7) ^ fb) << 4);
    }
#define STAGE_B(POS) do {                                                                                        \
      const unsigned base_ = lds0 + LDS_B + (unsigned)((POS) & 3) * B_BYTES + (unsigned)w * 4096;                \
      const size_t go_ = (size_t)(POS) * (BK * NS);                                                              \
      _Pragma("unroll") for (int i = 0; i < 4; ++i) glds16(bptr[i] + go_, base_ + i * 1024);                     \
    } while (0)
    STAGE_B(0); STAGE_B(1); STAGE_B(2); STAGE_B(3);
    if (w == 1 && lane < 16) {
      unsigned sd = 0, r = 0;
#pragma unroll
      for (int j = 0; j < 4; ++j)
        if ((lane >> j) & 1) { sd |= (4u + r) << (8 * j); ++r; }
      misc[208 + lane] = sd;
    }
    asm volatile("s_waitcnt vmcnt(16) lgkmcnt(0)" : "+v"(seq), "+v"(sxv), "+v"(swv) :: "memory");
    seq = __builtin_amdgcn_readfirstlane(seq) + 1u;
    const float scale = sxv * swv;
    if (w == 0 && grp != 0) {
      const unsigned q = seq & 1u, tag = (seq | 0x800000u) << 8;
      const unsigned* fptr = (const unsigned*)(myws + FLAG_OFF) + ((q * NR + src) * 8 + rb) * 128 + (lane < RAWK ? lane : 0) * 4;
      const unsigned long long t0 = __builtin_amdgcn_s_memrealtime();
      for (unsigned it = 1;; ++it) {
        u32x4 f;
        asm volatile("global_load_dwordx4 %0, %1, off sc0 sc1\n\ts_waitcnt vmcnt(0)" : "=v"(f) : "v"(fptr) : "memory");
        if (__all(tags_ok(f, tag))) {
          const unsigned char* g = p.ws[src] + EXPR_OFF + (size_t)q * SHARD_BYTES + (size_t)(rb * 4 + cb) * SEC_RAW + lane * 16;
#pragma unroll
          for (int k = 0; k < RAWK; ++k) {
            glds16(g + (size_t)k * (8 * 8192), lds0 + LDS_MISC + 1280);
            glds16(g + (size_t)k * (8 * 8192) + 1024, lds0 + LDS_MISC + 1280);
          }
          break;
        }
        if ((it & 63u) == 0) {
          if (__hip_atomic_load(tmo, __ATOMIC_RELAXED, __HIP_MEMORY_SCOPE_AGENT) == seq ||
              __builtin_amdgcn_s_memrealtime() - t0 > DEADLINE_TICKS) break;
        }
        if (it > 64u) __builtin_amdgcn_s_sleep(8);
      }
    } else
    WAIT_VM(8);
    const int wm = w >> 1, wn = w & 1, fr = lane & 15, g = lane >> 4;
    const unsigned xo0 = (32 * wm + fr) * 128 + ((g ^ ((fr >> 1) & 7)) << 4);
    const unsigned xo1 = (32 * wm + fr) * 128 + (((g + 4) ^ ((fr >> 1) & 7)) << 4);
    unsigned wo[4];
#pragma unroll
    for (int nf = 0; nf < 4; ++nf)
      wo[nf] = (16 * g + (fr >> 1)) * 128 + (((4 * wn + nf) ^ (((fr >> 2) & 3) | ((g & 1) << 2))) << 4) + 8 * (fr & 1);
    float* orow = p.out + (size_t)(src * MS + rb * BM + 32 * wm + fr) * NS + cb * BN + 64 * wn + 4 * g;
    asm volatile("" : "+v"(orow));
    v4f acc[4][2];
#pragma unroll
    for (int nf = 0; nf < 4; ++nf)
#pragma unroll
      for (int mf = 0; mf < 2; ++mf) acc[nf][mf] = (v4f){0.f, 0.f, 0.f, 0.f};
    v8i wf[4];
#define READ_W(POS) do {                                                                                         \
      const unsigned char* sb_ = smem + LDS_B + ((POS) & 3) * B_BYTES;                                           \
      _Pragma("unroll") for (int nf = 0; nf < 4; ++nf) {                                                         \
        v2i t_[4];                                                                                               \
        _Pragma("unroll") for (int c = 0; c < 4; ++c)                                                            \
          t_[c] = __builtin_amdgcn_ds_read_tr8_b64_v2i32((LAS v2i*)(sb_ + wo[nf] + (64 * (c >> 1) + 8 * (c & 1)) * 128)); \
        wf[nf] = (v8i){t_[0].x, t_[0].y, t_[1].x, t_[1].y, t_[2].x, t_[2].y, t_[3].x, t_[3].y};                  \
      }                                                                                                          \
    } while (0)
    __builtin_amdgcn_s_barrier();
    const bool anybad = misc[4] != 0;
    READ_W(0);
    WAIT_LGKM();
    __builtin_amdgcn_s_barrier();
#pragma unroll 1
    for (int k = 1; k <= NSTEP; ++k) {
      const unsigned char* sa = smem + LDS_A + ((k - 1) & 1) * A_BYTES;
      v8i xf[2];
#pragma unroll
      for (int mf = 0; mf < 2; ++mf) {
        const v4i l_ = *(const v4i*)(sa + xo0 + mf * 2048);
        const v4i h_ = *(const v4i*)(sa + xo1 + mf * 2048);
        xf[mf] = (v8i){l_.x, l_.y, l_.z, l_.w, h_.x, h_.y, h_.z, h_.w};
      }
      if (k + 3 < NSTEP) STAGE_B(k + 3);
#pragma unroll
      for (int mf = 0; mf < 2; ++mf)
#pragma unroll
        for (int nf = 0; nf < 4; ++nf)
          acc[nf][mf] = __builtin_amdgcn_mfma_scale_f32_16x16x128_f8f6f4(wf[nf], xf[mf], acc[nf][mf], 1, 0, 0, 127, 0, 127);
      asm volatile("" : "+v"(acc[0][0]), "+v"(acc[0][1]), "+v"(acc[1][0]), "+v"(acc[1][1]),
                        "+v"(acc[2][0]), "+v"(acc[2][1]), "+v"(acc[3][0]), "+v"(acc[3][1]));
      if (k == NSTEP) break;
      READ_W(k);
      if (k <= NSTEP - 4) WAIT_VM(8);
      else if (k == NSTEP - 3) WAIT_VM(4);
      else WAIT_VM(0);
      WAIT_LGKM();
      __builtin_amdgcn_s_barrier();
    }
    TS(39);
    const float sc = anybad ? __builtin_nanf("") : scale;
    const v4f zero4 = {0.f, 0.f, 0.f, 0.f};
#pragma unroll
    for (int mf = 0; mf < 2; ++mf) {
      const v4f v0 = __builtin_elementwise_maximum(acc[0][mf] * sc, zero4);
      const v4f v1 = __builtin_elementwise_maximum(acc[1][mf] * sc, zero4);
      const v4f v2 = __builtin_elementwise_maximum(acc[2][mf] * sc, zero4);
      const v4f v3 = __builtin_elementwise_maximum(acc[3][mf] * sc, zero4);
      asm volatile("global_store_dwordx4 %0, %1, off sc1\n\tglobal_store_dwordx4 %0, %2, off offset:64 sc1\n\t"
                   "global_store_dwordx4 %0, %3, off offset:128 sc1\n\tglobal_store_dwordx4 %0, %4, off offset:192 sc1\n\ts_nop 1"
                   :: "v"(orow + mf * 16 * NS), "v"(v0), "v"(v1), "v"(v2), "v"(v3) : "memory");
    }
    return;
  }

  const unsigned rowoff = (unsigned)(16 * w + (lane >> 2)) * KD + (lane & 3) * 16;
  const unsigned char* xp = p.x + (size_t)(grp * BM) * KD + idx * BK + rowoff;
  unsigned char* peerws;
  const unsigned char* pwp = (const unsigned char*)__builtin_amdgcn_kernarg_segment_ptr() + offsetof(Params, ws) +
                             ((unsigned)(rank + lane) & 7u) * 8;
  u32x4 xv0, xv1;
  unsigned seq;
  asm volatile("global_load_dwordx4 %0, %4, off\n\tglobal_load_dwordx4 %1, %4, off offset:64\n\tglobal_load_dword %2, %5, off sc1\n\t"
               "global_load_dwordx2 %3, %6, off"
               : "=&v"(xv0), "=&v"(xv1), "=&v"(seq), "=&v"(peerws) : "v"(xp), "v"(seqp), "v"(pwp) : "memory");
  __builtin_amdgcn_s_barrier();
  LAS unsigned* selc = misc + 224 + 16 * w;
  const unsigned selc_addr = lds0 + LDS_MISC + 896 + 64 * w;
  if (lane < 16) {
    unsigned sc = 0x0C0C0C0Cu, r = 0;
#pragma unroll
    for (int j = 0; j < 4; ++j)
      if ((lane >> j) & 1) { sc = (sc & ~(0xFFu << (8 * r))) | ((unsigned)j << (8 * r)); ++r; }
    selc[lane] = sc;
  }
  asm volatile("s_waitcnt vmcnt(0)" : "+v"(xv0), "+v"(xv1), "+v"(seq), "+v"(peerws) :: "memory");
  TS(42);
  seq = __builtin_amdgcn_readfirstlane(seq) + 1u;
  const unsigned q = seq & 1u;
  const unsigned tag = (seq | 0x800000u) << 8;
  unsigned meta = 0;
  if (idx < RAWK || idx >= NSTEP - RAWL) {
    st16x2_sys(myws + EXPR_OFF + (size_t)q * SHARD_BYTES + (size_t)(b * 4 + w) * SEC_RAW + lane * 16, xv0, xv1);
  } else {
    unsigned D[8] = {xv0.x, xv0.y, xv0.z, xv0.w, xv1.x, xv1.y, xv1.z, xv1.w};
    LAS unsigned char* scr = (LAS unsigned char*)smem + LDS_A + w * 2048;
    constexpr int NIB = A_BYTES;
    unsigned base, b3;
    {
      unsigned sum = 0;
#pragma unroll
      for (int i = 0; i < 8; ++i) sum = __builtin_amdgcn_sad_u8(D[i] & 0x78787878u, 0u, sum);
      const int tot = (int)__builtin_amdgcn_readlane(wave_scan(sum), 63);
      const int t = (tot - 54067) >> 14, t3 = (tot - 819) >> 14;
      base = t < 0 ? 0u : (t > 8 ? 8u : (unsigned)t);
      b3 = t3 < 0 ? 0u : (t3 > 12 ? 12u : (unsigned)t3);
    }
    bool done = false;
    {
      if (lane < 60) *(LAS unsigned*)(scr + NIB + lane * 4) = 0u;
      const unsigned k1 = (128u - 8u * b3) * 0x01010101u, k3 = (104u - 8u * b3) * 0x01010101u;
      const unsigned kLo = (b3 >= 6u ? 176u - 8u * b3 : 128u) * 0x01010101u;
      const unsigned kHi = (b3 <= 10u ? 88u - 8u * b3 : 0u) * 0x01010101u;
      const unsigned kC = (64u + ((48u - 8u * b3) & 63u)) * 0x01010101u;
      unsigned C[8], c2[8], sel[8], run[9], far = 0;
      run[0] = 0;
#pragma unroll
      for (int i = 0; i < 8; ++i) {
        const unsigned u = D[i] & 0x78787878u;
        const unsigned g1 = u + k1, g3 = u + k3;
        const unsigned esc = (~g1 | g3) & 0x80808080u, h = g3 & 0x80808080u;
        far |= ~(u + kLo) | (u + kHi);
        C[i] = (D[i] & 0x87878787u) | (g1 & 0x18181818u) | ((esc - (esc >> 2)) >> 2);
        c2[i] = ((u + kC - ((h - (h >> 2)) >> 2)) >> 3) & 0x07070707u;
        sel[i] = lds_u32(__builtin_amdgcn_udot4(esc >> 7, 0x20100804u, selc_addr, false));
        run[i + 1] = __builtin_amdgcn_udot4(esc >> 7, 0x03030303u, run[i], false);
      }
      const unsigned cnt = run[8];
      const unsigned inc = wave_scan(cnt);
      const unsigned total = __builtin_amdgcn_readlane(inc, 63);
      if (total <= 1791u && !__any((far & 0x80808080u) != 0u || cnt > 63u)) {
        unsigned long long acc = 0;
#pragma unroll
        for (int i = 0; i < 8; ++i) {
          const unsigned y = __builtin_amdgcn_perm(0u, c2[i], sel[i]);
          const unsigned v = __builtin_amdgcn_udot4(y, 0x00400801u, 0u, false) + ((y >> 24) << 9);
          acc |= (unsigned long long)v << run[i];
        }
#pragma unroll
        for (int r = 0; r < 2; ++r) {
          C[3 * r] |= (C[6 + r] << 5) & 0x60606060u;
          C[3 * r + 1] |= (C[6 + r] << 3) & 0x60606060u;
          C[3 * r + 2] |= ((C[6 + r] >> 1) & 0x40404040u) | ((C[6 + r] << 1) & 0x20202020u);
        }
        const unsigned bp = inc - cnt, sh = bp & 31u;
        LAS unsigned* sp = (LAS unsigned*)(scr + NIB) + (bp >> 5);
        const unsigned long long a0 = acc << sh;
        __hip_atomic_fetch_or(sp, (unsigned)a0, __ATOMIC_RELAXED, __HIP_MEMORY_SCOPE_WORKGROUP);
        __hip_atomic_fetch_or(sp + 1, (unsigned)(a0 >> 32), __ATOMIC_RELAXED, __HIP_MEMORY_SCOPE_WORKGROUP);
        __hip_atomic_fetch_or(sp + 2, (unsigned)((acc >> 32) << sh >> 32), __ATOMIC_RELAXED, __HIP_MEMORY_SCOPE_WORKGROUP);
        const unsigned stv = *(volatile LAS unsigned*)(scr + NIB + (lane < 56 ? lane : 55) * 4);
        unsigned char* sec = myws + EXP6_OFF + (size_t)q * SHARD_BYTES + (size_t)b * BLK_C6 + w * SEC_C6;
        typedef unsigned u32x2 __attribute__((ext_vector_type(2)));
        const u32x4 c0 = {C[0], C[1], C[2], C[3]};
        const u32x2 c1 = {C[4], C[5]};
        asm volatile("global_store_dwordx4 %0, %3, off sc0 sc1\n\tglobal_store_dwordx2 %1, %4, off sc0 sc1\n\t"
                     "global_store_dword %2, %5, off sc0 sc1\n\ts_nop 1"
                     :: "v"(sec + lane * 16), "v"(sec + 1024 + lane * 8), "v"(sec + 1536 + (lane < 56 ? lane : 55) * 4),
                        "v"(c0), "v"(c1), "v"(stv) : "memory");
        meta = 0x40u | b3;
        done = true;
      }
    }
    if (!done) {
    *(LAS u32x4*)(scr + lane * 32) = xv0;
    *(LAS u32x4*)(scr + lane * 32 + 16) = xv1;
    if (lane < 16) *(LAS unsigned*)(scr + NIB + lane * 4) = 0u;
    const unsigned cB = (128u - 8u * base) * 0x01010101u, cB7 = (72u - 8u * base) * 0x01010101u;
    const unsigned cLo = (base >= 6u ? 176u - 8u * base : 128u) * 0x01010101u;
    const unsigned cHi = (base <= 6u ? 56u - 8u * base : 0u) * 0x01010101u;
    unsigned C[8], M = 0, far = 0;
#pragma unroll
    for (int i = 0; i < 8; ++i) {
      const unsigned u = D[i] & 0x78787878u;
      const unsigned g1 = u + cB, g2 = u + cB7;
      const unsigned esc = (~g1 | g2) & 0x80808080u;
      far |= ~(u + cLo) | (u + cHi);
      C[i] = (D[i] & 0x87878787u) | (g1 & 0x38383838u) | ((esc - (esc >> 3)) >> 1);
      M |= esc >> (7 - i);
    }
#pragma unroll
    for (int i = 0; i < 7; ++i) C[i] |= shift_pos(C[7], CPOS[i], 6) & 0x40404040u;
    const unsigned cnt = __builtin_popcount(M);
    const unsigned inc = wave_scan(cnt);
    const unsigned total = __builtin_amdgcn_readlane(inc, 63);
    const bool use3 = total <= 85u && !__any((far & 0x80808080u) != 0u);
    if ((use3 || total <= 64u) && !__any(cnt > 8u)) {
      const unsigned wd = use3 ? 3u : 4u;
      unsigned S = 0, Mm = M;
      unsigned by[8];
#pragma unroll
      for (int k = 0; k < 8; ++k) {
        const unsigned bt = (unsigned)__builtin_ctz(Mm | 0x80000000u);
        by[k] = *(LAS unsigned char*)(scr + lane * 32 + ((bt & 7) << 2) + (bt >> 3));
        Mm &= Mm - 1;
      }
#pragma unroll
      for (int k = 0; k < 8; ++k) {
        const unsigned e = (by[k] >> 3) & 15u;
        const int d = (int)e - (int)base;
        const unsigned code = use3 ? (unsigned)(d < 0 ? -1 - d : d - 1) : e;
        S |= (k < (int)cnt ? code : 0u) << (wd * k);
      }
      const unsigned bp = (inc - cnt) * wd, sh = bp & 31u;
      if (cnt) {
        LAS unsigned* sp = (LAS unsigned*)(scr + NIB) + (bp >> 5);
        __hip_atomic_fetch_or(sp, S << sh, __ATOMIC_RELAXED, __HIP_MEMORY_SCOPE_WORKGROUP);
        __hip_atomic_fetch_or(sp + 1, (unsigned)(((unsigned long long)S << sh) >> 32), __ATOMIC_RELAXED, __HIP_MEMORY_SCOPE_WORKGROUP);
      }
      const unsigned stv = *(volatile LAS unsigned*)(scr + NIB + (lane < 8 ? lane : 7) * 4);
      unsigned char* sec = myws + EXPC_OFF + (size_t)q * SHARD_BYTES + (size_t)b * BLK_CMP + w * SEC_CMP;
      typedef unsigned u32x3 __attribute__((ext_vector_type(3)));
      const u32x4 c0 = {C[0], C[1], C[2], C[3]};
      const u32x3 c1 = {C[4], C[5], C[6]};
      asm volatile("global_store_dwordx4 %0, %3, off sc0 sc1\n\tglobal_store_dwordx3 %1, %4, off sc0 sc1\n\t"
                   "global_store_dword %2, %5, off sc0 sc1\n\ts_nop 1"
                   :: "v"(sec + lane * 16), "v"(sec + 1024 + lane * 12), "v"(sec + 1792 + (lane < 8 ? lane : 7) * 4),
                      "v"(c0), "v"(c1), "v"(stv) : "memory");
      meta = (use3 ? 0xC0u : 0x80u) | base;
    } else {
      st16x2_sys(myws + EXPR_OFF + (size_t)q * SHARD_BYTES + (size_t)(b * 4 + w) * SEC_RAW + lane * 16, xv0, xv1);
    }
    }
  }
  TS(1);
  WAIT_VM(0);
  if (lane == 0) *(volatile LAS unsigned*)(misc + w) = tag | meta;
  TS(2);
  if (w == 3) {
    u32x4 fv;
    do {
      asm volatile("ds_read_b128 %0, %1\n\ts_waitcnt lgkmcnt(0)" : "=v"(fv) : "v"(lds0 + LDS_MISC) : "memory");
    } while (fv.x == 0u || fv.y == 0u || fv.z == 0u || fv.w == 0u);
    if (lane >= 1 && lane < NR)
      asm volatile("global_store_dwordx4 %0, %1, off sc0 sc1\n\ts_nop 1"
                   :: "v"((unsigned*)(peerws + FLAG_OFF) + ((q * NR + rank) * 8 + grp) * 128 + idx * 4), "v"(fv) : "memory");
  } else if (w == 0) {
    unsigned bad = 0;
    u32x4 f = {0u, 0u, 0u, 0u};
    if (grp != 0) {
      const unsigned* fptr = (const unsigned*)(myws + FLAG_OFF) + ((q * NR + src) * 8 + rb) * 128 + (lane & 31) * 4;
      const unsigned long long t0 = __builtin_amdgcn_s_memrealtime();
      for (unsigned it = 1;; ++it) {
        asm volatile("global_load_dwordx4 %0, %1, off sc0 sc1\n\ts_waitcnt vmcnt(0)" : "=v"(f) : "v"(fptr) : "memory");
        if (__all(tags_ok(f, tag))) break;
        if (it > 64u) __builtin_amdgcn_s_sleep(8);
        if ((it & 63u) == 0 && (__hip_atomic_load(tmo, __ATOMIC_RELAXED, __HIP_MEMORY_SCOPE_AGENT) == seq ||
                                __builtin_amdgcn_s_memrealtime() - t0 > DEADLINE_TICKS)) {
          if (lane == 0) __hip_atomic_store(tmo, seq, __ATOMIC_RELAXED, __HIP_MEMORY_SCOPE_AGENT);
          bad = 1;
          f = (u32x4){0u, 0u, 0u, 0u};
          break;
        }
      }
    }
    if (lane < 32) *(LAS u32x4*)(misc + 64 + lane * 4) = f;
    if (lane == 0) misc[4] = bad;
    WAIT_LGKM();
  }
  __builtin_amdgcn_s_barrier();
  if (w == 3 && lane == 0) __hip_atomic_store(seqp, seq, __ATOMIC_RELAXED, __HIP_MEMORY_SCOPE_AGENT);
  const unsigned flags = misc[64 + (lane & 31) * 4 + w];
  TS(3);

  const unsigned o1 = grp == 0 ? rowoff : (unsigned)lane * 16;
  const unsigned o2 = grp == 0 ? rowoff + 64 : 1024u + lane * 16;
  const unsigned char* raw_base = grp == 0 ? p.x + (size_t)(rb * BM) * KD
                                           : p.ws[src] + EXPR_OFF + (size_t)q * SHARD_BYTES + (size_t)(rb * 4 + w) * SEC_RAW;
  const unsigned raw_step = grp == 0 ? BK : 8 * 8192;
  const unsigned char* cmp_base = p.ws[src] + EXPC_OFF + (size_t)q * SHARD_BYTES + (size_t)rb * BLK_CMP + w * SEC_CMP;
  const unsigned char* c6_base = p.ws[src] + EXP6_OFF + (size_t)q * SHARD_BYTES + (size_t)rb * BLK_C6 + w * SEC_C6;
  const unsigned o2c = 1024u + lane * 12, o3c = 1792u + (lane < 8 ? lane : 7) * 4;
  const unsigned o2s = 1024u + lane * 8, o3s = 1536u + (lane < 56 ? lane : 55) * 4;
#define LOAD_A(LO, HI, ST, ME, POS) do {                                                                         \
    const int ks_ = (POS);                                                                                       \
    ME = __builtin_amdgcn_readlane(flags, ks_) & 0xFFu;                                                          \
    const bool c_ = (ME & 0x80u) != 0, s_ = (ME & 0xC0u) == 0x40u;                                               \
    const unsigned char* base_ = (s_ ? c6_base : (c_ ? cmp_base : raw_base)) +                                   \
                                 (unsigned)ks_ * (s_ ? 8u * BLK_C6 : (c_ ? 8u * BLK_CMP : raw_step));            \
    const unsigned oa_ = s_ ? o2s : (c_ ? o2c : o2), ob_ = s_ ? o3s : (c_ ? o3c : o1);                           \
    asm volatile("global_load_dwordx4 %0, %3, %6\n\tglobal_load_dwordx4 %1, %4, %6\n\tglobal_load_dword %2, %5, %6" \
                 : "=&v"(LO), "=&v"(HI), "=&v"(ST) : "v"(o1), "v"(oa_), "v"(ob_), "s"(base_) : "memory");        \
  } while (0)
  u32x4 lo0, hi0, lo1, hi1;
  unsigned st0, st1, me0, me1;
  LOAD_A(lo0, hi0, st0, me0, 0);
  LOAD_A(lo1, hi1, st1, me1, 1);
  unsigned awr0, awr1;
  {
    const int r = 16 * w + (lane >> 2), fa = (r >> 1) & 7;
    awr0 = r * 128 + (((lane & 3) ^ fa) << 4);
    awr1 = r * 128 + ((((lane & 3) + 4) ^ fa) << 4);
  }
#define STEP(LO, HI, ST, ME, I, U) do {                                                                          \
    const int i = (I);                                                                                           \
    if (i == 16) TS(44);                                                                                         \
    if (i < NSTEP - 1) WAIT_VM(3); else WAIT_VM(0);                                                              \
    asm volatile("" : "+v"(LO), "+v"(HI), "+v"(ST));                                                             \
    TS(4 + i);                                                                                                   \
    unsigned char* sa = smem + LDS_A + (U) * A_BYTES;                                                            \
    if ((ME & 0xC0u) == 0x40u) {                                                                                 \
      const unsigned b3 = (ME & 15u) * 0x01010101u;                                                              \
                          \
      const unsigned LL = (((b3 + 0x0D0C0B0Au) & 0x0F0F0F0Fu) ^ (b3 + 0x03030303u)) << 3;                        \
      const unsigned LH = (((b3 + 0x14130F0Eu) & 0x0F0F0F0Fu) ^ (b3 + 0x03030303u)) << 3;                        \
      unsigned C[8] = {LO.x, LO.y, LO.z, LO.w, HI.x, HI.y, 0u, 0u};                                              \
      _Pragma("unroll") for (int r = 0; r < 2; ++r)                                                             \
        C[6 + r] = ((C[3 * r] >> 5) & 0x03030303u) | ((C[3 * r + 1] >> 3) & 0x0C0C0C0Cu) |                       \
                   ((C[3 * r + 2] << 1) & 0x80808080u) | ((C[3 * r + 2] >> 1) & 0x10101010u);                    \
      unsigned sel[8], run[9];                                                                                   \
      run[0] = 0;                                                                                                \
      _Pragma("unroll") for (int k = 0; k < 8; ++k) {                                                            \
        C[k] &= 0x9F9F9F9Fu;                                                                                     \
        const unsigned f = ((C[k] + 0x08080808u) >> 5) & 0x01010101u;                                            \
        sel[k] = lds_u32(__builtin_amdgcn_udot4(f, 0x20100804u, lds0 + LDS_MISC + 832, false));                  \
        run[k + 1] = __builtin_amdgcn_udot4(f, 0x03030303u, run[k], false);                          \
      }                                                                                                          \
      const unsigned bp = wave_scan(run[8]) - run[8];                                                            \
      const unsigned w0 = __builtin_amdgcn_ds_bpermute((int)((bp >> 5) << 2), (int)ST);                          \
      const unsigned w1 = __builtin_amdgcn_ds_bpermute((int)(((bp >> 5) + 1) << 2), (int)ST);                    \
      const unsigned w2 = __builtin_amdgcn_ds_bpermute((int)(((bp >> 5) + 2) << 2), (int)ST);                    \
      const unsigned long long W = ((unsigned long long)__builtin_amdgcn_alignbit(w2, w1, bp & 31u) << 32) |     \
                                   __builtin_amdgcn_alignbit(w1, w0, bp & 31u);                                  \
      _Pragma("unroll") for (int k = 0; k < 8; ++k) {                                                            \
        const unsigned v = (unsigned)(W >> run[k]);                                                              \
        const unsigned y = (((v & 0x1C7u) * 0x401u) & 0x00070007u) |                  \
                           (((v & 0xE38u) * 0x8020u) & 0x07000700u);                  \
        const unsigned z = __builtin_amdgcn_perm(y, 0x0C0C0C0Cu, sel[k]);                                        \
        C[k] = (C[k] + (b3 << 3)) ^ __builtin_amdgcn_perm(LH, LL, z);                                            \
      }                                                                                                          \
      *(u32x4*)(sa + awr0) = (u32x4){C[0], C[1], C[2], C[3]};                                                    \
      *(u32x4*)(sa + awr1) = (u32x4){C[4], C[5], C[6], C[7]};                                                    \
    } else if (ME & 0x80u) {                                                                                     \
      const unsigned base = ME & 15u;                                                                            \
      unsigned C[8] = {LO.x, LO.y, LO.z, LO.w, HI.x, HI.y, HI.z, 0u};                                            \
      _Pragma("unroll") for (int k = 0; k < 7; ++k) C[7] |= shift_pos(C[k] & 0x40404040u, 6, CPOS[k]);           \
      unsigned M = 0, Dd[8];                                                                                     \
      _Pragma("unroll") for (int k = 0; k < 8; ++k) {                                                            \
        const unsigned t = C[k] & 0xBFBFBFBFu;                                                                   \
        M |= shift_pos((t + 0x08080808u) & 0x40404040u, 6, k);                                                   \
        Dd[k] = t + base * 0x08080808u;                                                                          \
      }                                                                                                          \
      *(u32x4*)(sa + awr0) = (u32x4){Dd[0], Dd[1], Dd[2], Dd[3]};                                                \
      *(u32x4*)(sa + awr1) = (u32x4){Dd[4], Dd[5], Dd[6], Dd[7]};                                                \
      const unsigned cnt = __builtin_popcount(M);                                                                \
      const bool use3 = (ME & 0x40u) != 0;                                                                       \
      const unsigned wd = use3 ? 3u : 4u;                                                                        \
      const unsigned bp = (wave_scan(cnt) - cnt) * wd;                                                           \
      const unsigned w0 = __builtin_amdgcn_ds_bpermute((int)((bp >> 5) << 2), (int)ST);                          \
      const unsigned w1 = __builtin_amdgcn_ds_bpermute((int)(((bp >> 5) + 1) << 2), (int)ST);                    \
      unsigned S = __builtin_amdgcn_alignbit(w1, w0, bp & 31u);                                                  \
                          \
      const unsigned abase = (unsigned)(uintptr_t)(LAS unsigned char*)sa + awr0;                                 \
      if (use3) {                                                                                                \
        while (__any(M != 0u)) {                                                                                 \
          const unsigned bt = (unsigned)__builtin_ctz(M | 0x80000000u);                                          \
          const unsigned c = S & 7u;                                                                             \
          unsigned dl = c < 6u ? 0u - 8u - c : c - 6u;                                                           \
          dl = M ? dl : 0u;                                                                                      \
          M &= M - 1;                                                                                            \
          S >>= 3;                                                                                               \
          asm volatile("ds_add_u32 %0, %1" :: "v"(abase ^ (((bt << 4) & 64u) | ((bt << 2) & 12u))),              \
                       "v"(dl << ((bt & 24u) | 3u)) : "memory");                                                 \
        }                                                                                                        \
      } else {                                                                                                   \
        while (__any(M != 0u)) {                                                                                 \
          const unsigned bt = (unsigned)__builtin_ctz(M | 0x80000000u);                                          \
          const unsigned dl = M ? (S & 15u) - base - 7u : 0u;                                                    \
          M &= M - 1;                                                                                            \
          S >>= 4;                                                                                               \
          asm volatile("ds_add_u32 %0, %1" :: "v"(abase ^ (((bt << 4) & 64u) | ((bt << 2) & 12u))),              \
                       "v"(dl << ((bt & 24u) | 3u)) : "memory");                                                 \
        }                                                                                                        \
      }                                                                                                          \
    } else {                                                                                                     \
      *(u32x4*)(sa + awr0) = LO;                                                                                 \
      *(u32x4*)(sa + awr1) = HI;                                                                                 \
    }                                                                                                            \
    WAIT_LGKM();                                                               \
    if (i + 2 < NSTEP) LOAD_A(LO, HI, ST, ME, i + 2);                                                            \
    if (i == 16) TS(45);                                                                                         \
    __builtin_amdgcn_s_barrier();                                                                         \
    if (i == 16) TS(46);                                                                                         \
  } while (0)
#pragma unroll 1
  for (int ii = 0; ii < NSTEP; ii += 2) {
    STEP(lo0, hi0, st0, me0, ii, 0);
    STEP(lo1, hi1, st1, me1, ii + 1, 1);
  }
  TS_DUMP();
}

}

extern "C" void kernel_launch(int rank, int nranks, ncclComm_t comm,
                              hipStream_t stream,
                              void* const* d_in, const int* in_sizes, int n_in,
                              void* d_out, int out_size,
                              const int* in_dtypes, int out_dtype,
                              void* const* d_out_all,
                              void* const* d_ws_all, size_t ws_bytes) {
  if (nranks != NR || ws_bytes < WS_NEED || n_in != 4) { fprintf(stderr, "unexpected configuration\n"); return; }
  HIP_CHECK(hipFuncSetAttribute((const void*)ag_gemm_kernel, hipFuncAttributeMaxDynamicSharedMemorySize, LDS_BYTES));
  Params p;
  p.x = (const unsigned char*)d_in[0];
  p.w = (const unsigned char*)d_in[1];
  p.sx = (const float*)d_in[2];
  p.sw = (const float*)d_in[3];
  p.out = (float*)d_out;
  for (int i = 0; i < NR; ++i) p.ws[i] = (unsigned char*)d_ws_all[i];
  p.myws = (unsigned char*)d_ws_all[rank];
  p.rank = rank;
  ag_gemm_kernel<<<NWG, NTHR, LDS_BYTES, stream>>>(p);
  HIP_CHECK(hipGetLastError());
}
```

```cpp
#include <hip/hip_runtime.h>
#include <rccl/rccl.h>
#include <cstdio>
#include <cstdint>
#include <cstddef>

#define HIP_CHECK(expr) do { hipError_t _e = (expr); if (_e != hipSuccess) { \
    fprintf(stderr, "HIP error %s at %s:%d\n", hipGetErrorString(_e), __FILE__, __LINE__); \
    return; } } while (0)

namespace {

typedef int v2i __attribute__((ext_vector_type(2)));
typedef int v4i __attribute__((ext_vector_type(4)));
typedef int v8i __attribute__((ext_vector_type(8)));
typedef float v4f __attribute__((ext_vector_type(4)));
typedef unsigned u32x4 __attribute__((ext_vector_type(4)));
#define LAS __attribute__((address_space(3)))

#define RAWK 1
#define RAWL 2

constexpr int NR = 8;
constexpr int MS = 512;
constexpr int KD = 4096;
constexpr int NS = 512;
constexpr int BM = 64, BN = 128, BK = 128, NSTEP = KD / BK;
constexpr int A_BYTES = BM * BK, B_BYTES = BK * BN;
constexpr int LDS_B = 0, LDS_A = 4 * B_BYTES, LDS_MISC = LDS_A + 2 * A_BYTES;
constexpr int LDS_BYTES = LDS_MISC + 2304;
constexpr int NWG = 256;
constexpr int NTHR = 512;
constexpr unsigned SEC_RAW = 2048;
constexpr unsigned SEC_CMP = 1824;
constexpr unsigned BLK_CMP = 7296;
constexpr unsigned SEC_C6 = 1760;
constexpr unsigned BLK_C6 = 7040;
constexpr size_t SHARD_BYTES = (size_t)MS * KD;
constexpr size_t EXPR_OFF = 0;
constexpr size_t EXPC_OFF = 2 * SHARD_BYTES;
constexpr size_t EXP6_OFF = 4 * SHARD_BYTES;
constexpr size_t FLAG_OFF = 6 * SHARD_BYTES;
constexpr size_t SEQ_OFF = FLAG_OFF + 2 * NR * 4096;
constexpr size_t TMO_OFF = SEQ_OFF + 8192;
constexpr size_t WS_NEED = TMO_OFF + 256;
constexpr unsigned long long DEADLINE_TICKS = 100000000ull;

struct Params {
  const unsigned char* x;
  const unsigned char* w;
  const float* sx;
  const float* sw;
  float* out;
  unsigned char* myws;
  unsigned char* ws[NR];
  int rank;
};

__device__ __forceinline__ void glds16(const void* g, unsigned lds_addr) {
  unsigned keep;
  asm volatile("s_mov_b32 %0, m0\n\ts_mov_b32 m0, %2\n\ts_nop 0\n\tglobal_load_lds_dwordx4 %1, off\n\ts_mov_b32 m0, %0"
               : "=&s"(keep) : "v"(g), "s"(lds_addr) : "memory");
}
__device__ __forceinline__ void st16x2_sys(void* p, u32x4 v0, u32x4 v1) {
  asm volatile("global_store_dwordx4 %0, %1, off sc0 sc1\n\tglobal_store_dwordx4 %0, %2, off offset:1024 sc0 sc1\n\ts_nop 1"
               :: "v"(p), "v"(v0), "v"(v1) : "memory");
}
#define WAIT_VM(n) asm volatile("s_waitcnt vmcnt(%0)" :: "i"(n) : "memory")
#define WAIT_LGKM() asm volatile("s_waitcnt lgkmcnt(0)" ::: "memory")
#define TS(i) do {} while (0)
#define TS_DUMP() do {} while (0)

__device__ __forceinline__ unsigned wave_scan(unsigned x) {
  x += __builtin_amdgcn_update_dpp(0u, x, 0x111, 0xf, 0xf, true);
  x += __builtin_amdgcn_update_dpp(0u, x, 0x112, 0xf, 0xf, true);
  x += __builtin_amdgcn_update_dpp(0u, x, 0x114, 0xf, 0xf, true);
  x += __builtin_amdgcn_update_dpp(0u, x, 0x118, 0xf, 0xf, true);
  x += __builtin_amdgcn_update_dpp(0u, x, 0x142, 0xa, 0xf, false);
  x += __builtin_amdgcn_update_dpp(0u, x, 0x143, 0xc, 0xf, false);
  return x;
}
__device__ __forceinline__ unsigned shift_pos(unsigned v, int from, int to) { return to >= from ? v << (to - from) : v >> (from - to); }
__device__ constexpr int CPOS[7] = {0, 1, 2, 3, 4, 5, 7};
__device__ __forceinline__ unsigned lds_u32(unsigned addr) { return *(LAS const unsigned*)(uintptr_t)addr; }
__device__ __forceinline__ bool tags_ok(u32x4 f, unsigned tag) { return (((f.x ^ tag) | (f.y ^ tag) | (f.z ^ tag) | (f.w ^ tag)) >> 8) == 0; }

__global__ __launch_bounds__(NTHR, 1) void ag_gemm_kernel(Params p) {
  extern __shared__ __attribute__((aligned(16))) unsigned char smem[];
  const int tid = threadIdx.x;
  const int lane = tid & 63;
  const int wv = __builtin_amdgcn_readfirstlane(tid >> 6);
  const int w = wv & 3;
  const int b = blockIdx.x;
  const int grp = b & 7, idx = b >> 3, rb = idx >> 2, cb = idx & 3;
  const int rank = p.rank;
  const unsigned src = (unsigned)(rank + grp) & 7u;
  unsigned char* myws = p.myws;
  asm volatile("" :: "s"(p.x), "s"(p.w), "s"(p.sx), "s"(p.sw), "s"(myws), "s"(rank));
  const unsigned lds0 = (unsigned)(uintptr_t)(LAS unsigned char*)smem;
  LAS unsigned* misc = (LAS unsigned*)(smem + LDS_MISC);
  unsigned* seqp = (unsigned*)(myws + SEQ_OFF) + b;
  unsigned* tmo = (unsigned*)(myws + TMO_OFF);
  TS(0);

  if (wv >= 4) {
    unsigned seq;
    float sxv, swv;
    asm volatile("global_load_dword %0, %3, off sc1\n\tglobal_load_dword %1, %4, off\n\tglobal_load_dword %2, %5, off"
                 : "=&v"(seq), "=&v"(sxv), "=&v"(swv) : "v"(seqp), "v"(p.sx), "v"(p.sw) : "memory");
    if (w == 0 && lane < 4) misc[lane] = 0u;
    WAIT_LGKM();
    __builtin_amdgcn_s_barrier();
    const unsigned char* bptr[4];
#pragma unroll
    for (int i = 0; i < 4; ++i) {
      const int kr = 32 * w + 8 * i + (lane >> 3);
      const int fb = ((kr >> 1) & 3) | (((kr >> 4) & 1) << 2);
      bptr[i] = p.w + (size_t)kr * NS + cb * BN + (((lane & 7) ^ fb) << 4);
    }
#define STAGE_B(POS) do {                                                                                        \
      const unsigned base_ = lds0 + LDS_B + (unsigned)((POS) & 3) * B_BYTES + (unsigned)w * 4096;                \
      const size_t go_ = (size_t)(POS) * (BK * NS);                                                              \
      _Pragma("unroll") for (int i = 0; i < 4; ++i) glds16(bptr[i] + go_, base_ + i * 1024);                     \
    } while (0)
    STAGE_B(0); STAGE_B(1); STAGE_B(2); STAGE_B(3);
    if (w == 1 && lane < 16) {
      unsigned sd = 0, r = 0;
#pragma unroll
      for (int j = 0; j < 4; ++j)
        if ((lane >> j) & 1) { sd |= (4u + r) << (8 * j); ++r; }
      misc[208 + lane] = sd;
    }
    asm volatile("s_waitcnt vmcnt(16) lgkmcnt(0)" : "+v"(seq), "+v"(sxv), "+v"(swv) :: "memory");
    seq = __builtin_amdgcn_readfirstlane(seq) + 1u;
    const float scale = sxv * swv;
    if (w == 0 && grp != 0) {
      const unsigned q = seq & 1u, tag = (seq | 0x800000u) << 8;
      const unsigned* fptr = (const unsigned*)(myws + FLAG_OFF) + ((q * NR + src) * 8 + rb) * 128 + (lane < RAWK ? lane : 0) * 4;
      const unsigned long long t0 = __builtin_amdgcn_s_memrealtime();
      for (unsigned it = 1;; ++it) {
        u32x4 f;
        asm volatile("global_load_dwordx4 %0, %1, off sc0 sc1\n\ts_waitcnt vmcnt(0)" : "=v"(f) : "v"(fptr) : "memory");
        if (__all(tags_ok(f, tag))) {
          const unsigned char* g = p.ws[src] + EXPR_OFF + (size_t)q * SHARD_BYTES + (size_t)(rb * 4 + cb) * SEC_RAW + lane * 16;
#pragma unroll
          for (int k = 0; k < RAWK; ++k) {
            glds16(g + (size_t)k * (8 * 8192), lds0 + LDS_MISC + 1280);
            glds16(g + (size_t)k * (8 * 8192) + 1024, lds0 + LDS_MISC + 1280);
          }
          break;
        }
        if ((it & 63u) == 0) {
          if (__hip_atomic_load(tmo, __ATOMIC_RELAXED, __HIP_MEMORY_SCOPE_AGENT) == seq ||
              __builtin_amdgcn_s_memrealtime() - t0 > DEADLINE_TICKS) break;
        }
        if (it > 64u) __builtin_amdgcn_s_sleep(8);
      }
    } else
    WAIT_VM(8);
    const int wm = w >> 1, wn = w & 1, fr = lane & 15, g = lane >> 4;
    const unsigned xo0 = (32 * wm + fr) * 128 + ((g ^ ((fr >> 1) & 7)) << 4);
    const unsigned xo1 = (32 * wm + fr) * 128 + (((g + 4) ^ ((fr >> 1) & 7)) << 4);
    unsigned wo[4];
#pragma unroll
    for (int nf = 0; nf < 4; ++nf)
      wo[nf] = (16 * g + (fr >> 1)) * 128 + (((4 * wn + nf) ^ (((fr >> 2) & 3) | ((g & 1) << 2))) << 4) + 8 * (fr & 1);
    float* orow = p.out + (size_t)(src * MS + rb * BM + 32 * wm + fr) * NS + cb * BN + 64 * wn + 4 * g;
    asm volatile("" : "+v"(orow));
    v4f acc[4][2];
#pragma unroll
    for (int nf = 0; nf < 4; ++nf)
#pragma unroll
      for (int mf = 0; mf < 2; ++mf) acc[nf][mf] = (v4f){0.f, 0.f, 0.f, 0.f};
    v8i wf[4];
#define READ_W(POS) do {                                                                                         \
      const unsigned char* sb_ = smem + LDS_B + ((POS) & 3) * B_BYTES;                                           \
      _Pragma("unroll") for (int nf = 0; nf < 4; ++nf) {                                                         \
        v2i t_[4];                                                                                               \
        _Pragma("unroll") for (int c = 0; c < 4; ++c)                                                            \
          t_[c] = __builtin_amdgcn_ds_read_tr8_b64_v2i32((LAS v2i*)(sb_ + wo[nf] + (64 * (c >> 1) + 8 * (c & 1)) * 128)); \
        wf[nf] = (v8i){t_[0].x, t_[0].y, t_[1].x, t_[1].y, t_[2].x, t_[2].y, t_[3].x, t_[3].y};                  \
      }                                                                                                          \
    } while (0)
    __builtin_amdgcn_s_barrier();
    const bool anybad = misc[4] != 0;
    READ_W(0);
    WAIT_LGKM();
    __builtin_amdgcn_s_barrier();
#pragma unroll 1
    for (int k = 1; k <= NSTEP; ++k) {
      const unsigned char* sa = smem + LDS_A + ((k - 1) & 1) * A_BYTES;
      v8i xf[2];
#pragma unroll
      for (int mf = 0; mf < 2; ++mf) {
        const v4i l_ = *(const v4i*)(sa + xo0 + mf * 2048);
        const v4i h_ = *(const v4i*)(sa + xo1 + mf * 2048);
        xf[mf] = (v8i){l_.x, l_.y, l_.z, l_.w, h_.x, h_.y, h_.z, h_.w};
      }
      if (k + 3 < NSTEP) STAGE_B(k + 3);
#pragma unroll
      for (int mf = 0; mf < 2; ++mf)
#pragma unroll
        for (int nf = 0; nf < 4; ++nf)
          acc[nf][mf] = __builtin_amdgcn_mfma_scale_f32_16x16x128_f8f6f4(wf[nf], xf[mf], acc[nf][mf], 1, 0, 0, 127, 0, 127);
      asm volatile("" : "+v"(acc[0][0]), "+v"(acc[0][1]), "+v"(acc[1][0]), "+v"(acc[1][1]),
                        "+v"(acc[2][0]), "+v"(acc[2][1]), "+v"(acc[3][0]), "+v"(acc[3][1]));
      if (k == NSTEP) break;
      READ_W(k);
      if (k <= NSTEP - 4) WAIT_VM(8);
      else if (k == NSTEP - 3) WAIT_VM(4);
      else WAIT_VM(0);
      WAIT_LGKM();
      __builtin_amdgcn_s_barrier();
    }
    TS(39);
    const float sc = anybad ? __builtin_nanf("") : scale;
    const v4f zero4 = {0.f, 0.f, 0.f, 0.f};
#pragma unroll
    for (int mf = 0; mf < 2; ++mf) {
      const v4f v0 = __builtin_elementwise_maximum(acc[0][mf] * sc, zero4);
      const v4f v1 = __builtin_elementwise_maximum(acc[1][mf] * sc, zero4);
      const v4f v2 = __builtin_elementwise_maximum(acc[2][mf] * sc, zero4);
      const v4f v3 = __builtin_elementwise_maximum(acc[3][mf] * sc, zero4);
      asm volatile("global_store_dwordx4 %0, %1, off sc1\n\tglobal_store_dwordx4 %0, %2, off offset:64 sc1\n\t"
                   "global_store_dwordx4 %0, %3, off offset:128 sc1\n\tglobal_store_dwordx4 %0, %4, off offset:192 sc1\n\ts_nop 1"
                   :: "v"(orow + mf * 16 * NS), "v"(v0), "v"(v1), "v"(v2), "v"(v3) : "memory");
    }
    return;
  }

  const unsigned rowoff = (unsigned)(16 * w + (lane >> 2)) * KD + (lane & 3) * 16;
  const unsigned char* xp = p.x + (size_t)(grp * BM) * KD + idx * BK + rowoff;
  unsigned char* peerws;
  const unsigned char* pwp = (const unsigned char*)__builtin_amdgcn_kernarg_segment_ptr() + offsetof(Params, ws) +
                             ((unsigned)(rank + lane) & 7u) * 8;
  u32x4 xv0, xv1;
  unsigned seq;
  asm volatile("global_load_dwordx4 %0, %4, off\n\tglobal_load_dwordx4 %1, %4, off offset:64\n\tglobal_load_dword %2, %5, off sc1\n\t"
               "global_load_dwordx2 %3, %6, off"
               : "=&v"(xv0), "=&v"(xv1), "=&v"(seq), "=&v"(peerws) : "v"(xp), "v"(seqp), "v"(pwp) : "memory");
  __builtin_amdgcn_s_barrier();
  LAS unsigned* selc = misc + 224 + 16 * w;
  const unsigned selc_addr = lds0 + LDS_MISC + 896 + 64 * w;
  if (lane < 16) {
    unsigned sc = 0x0C0C0C0Cu, r = 0;
#pragma unroll
    for (int j = 0; j < 4; ++j)
      if ((lane >> j) & 1) { sc = (sc & ~(0xFFu << (8 * r))) | ((unsigned)j << (8 * r)); ++r; }
    selc[lane] = sc;
  }
  asm volatile("s_waitcnt vmcnt(0)" : "+v"(xv0), "+v"(xv1), "+v"(seq), "+v"(peerws) :: "memory");
  TS(42);
  seq = __builtin_amdgcn_readfirstlane(seq) + 1u;
  const unsigned q = seq & 1u;
  const unsigned tag = (seq | 0x800000u) << 8;
  unsigned meta = 0;
  if (idx < RAWK || idx >= NSTEP - RAWL) {
    st16x2_sys(myws + EXPR_OFF + (size_t)q * SHARD_BYTES + (size_t)(b * 4 + w) * SEC_RAW + lane * 16, xv0, xv1);
  } else {
    unsigned D[8] = {xv0.x, xv0.y, xv0.z, xv0.w, xv1.x, xv1.y, xv1.z, xv1.w};
    LAS unsigned char* scr = (LAS unsigned char*)smem + LDS_A + w * 2048;
    constexpr int NIB = A_BYTES;
    unsigned base, b3;
    {
      unsigned sum = 0;
#pragma unroll
      for (int i = 0; i < 8; ++i) sum = __builtin_amdgcn_sad_u8(D[i] & 0x78787878u, 0u, sum);
      const int tot = (int)__builtin_amdgcn_readlane(wave_scan(sum), 63);
      const int t = (tot - 54067) >> 14, t3 = (tot - 819) >> 14;
      base = t < 0 ? 0u : (t > 8 ? 8u : (unsigned)t);
      b3 = t3 < 0 ? 0u : (t3 > 12 ? 12u : (unsigned)t3);
    }
    bool done = false;
    {
      if (lane < 60) *(LAS unsigned*)(scr + NIB + lane * 4) = 0u;
      const unsigned k1 = (128u - 8u * b3) * 0x01010101u, k3 = (104u - 8u * b3) * 0x01010101u;
      const unsigned kLo = (b3 >= 6u ? 176u - 8u * b3 : 128u) * 0x01010101u;
      const unsigned kHi = (b3 <= 10u ? 88u - 8u * b3 : 0u) * 0x01010101u;
      const unsigned kC = (64u + ((48u - 8u * b3) & 63u)) * 0x01010101u;
      unsigned C[8], c2[8], sel[8], run[9], far = 0;
      run[0] = 0;
#pragma unroll
      for (int i = 0; i < 8; ++i) {
        const unsigned u = D[i] & 0x78787878u;
        const unsigned g1 = u + k1, g3 = u + k3;
        const unsigned esc = (~g1 | g3) & 0x80808080u, h = g3 & 0x80808080u;
        far |= ~(u + kLo) | (u + kHi);
        C[i] = (D[i] & 0x87878787u) | (g1 & 0x18181818u) | ((esc - (esc >> 2)) >> 2);
        c2[i] = ((u + kC - ((h - (h >> 2)) >> 2)) >> 3) & 0x07070707u;
        sel[i] = lds_u32(__builtin_amdgcn_udot4(esc >> 7, 0x20100804u, selc_addr, false));
        run[i + 1] = __builtin_amdgcn_udot4(esc >> 7, 0x03030303u, run[i], false);
      }
      const unsigned cnt = run[8];
      const unsigned inc = wave_scan(cnt);
      const unsigned total = __builtin_amdgcn_readlane(inc, 63);
      if (total <= 1791u && !__any((far & 0x80808080u) != 0u || cnt > 63u)) {
        unsigned long long acc = 0;
#pragma unroll
        for (int i = 0; i < 8; ++i) {
          const unsigned y = __builtin_amdgcn_perm(0u, c2[i], sel[i]);
          const unsigned v = __builtin_amdgcn_udot4(y, 0x00400801u, 0u, false) + ((y >> 24) << 9);
          acc |= (unsigned long long)v << run[i];
        }
#pragma unroll
        for (int r = 0; r < 2; ++r) {
          C[3 * r] |= (C[6 + r] << 5) & 0x60606060u;
          C[3 * r + 1] |= (C[6 + r] << 3) & 0x60606060u;
          C[3 * r + 2] |= ((C[6 + r] >> 1) & 0x40404040u) | ((C[6 + r] << 1) & 0x20202020u);
        }
        const unsigned bp = inc - cnt, sh = bp & 31u;
        LAS unsigned* sp = (LAS unsigned*)(scr + NIB) + (bp >> 5);
        const unsigned long long a0 = acc << sh;
        __hip_atomic_fetch_or(sp, (unsigned)a0, __ATOMIC_RELAXED, __HIP_MEMORY_SCOPE_WORKGROUP);
        __hip_atomic_fetch_or(sp + 1, (unsigned)(a0 >> 32), __ATOMIC_RELAXED, __HIP_MEMORY_SCOPE_WORKGROUP);
        __hip_atomic_fetch_or(sp + 2, (unsigned)((acc >> 32) << sh >> 32), __ATOMIC_RELAXED, __HIP_MEMORY_SCOPE_WORKGROUP);
        const unsigned stv = *(volatile LAS unsigned*)(scr + NIB + (lane < 56 ? lane : 55) * 4);
        unsigned char* sec = myws + EXP6_OFF + (size_t)q * SHARD_BYTES + (size_t)b * BLK_C6;
        typedef unsigned u32x2 __attribute__((ext_vector_type(2)));
        const u32x4 c0 = {C[0], C[1], C[2], C[3]};
        const u32x2 c1 = {C[4], C[5]};
        asm volatile("global_store_dwordx4 %0, %3, off sc0 sc1\n\tglobal_store_dwordx2 %1, %4, off sc0 sc1\n\t"
                     "global_store_dword %2, %5, off sc0 sc1\n\ts_nop 1"
                     :: "v"(sec + w * 1024 + lane * 16), "v"(sec + 4096 + w * 512 + lane * 8),
                        "v"(sec + 6144 + w * 224 + (lane < 56 ? lane : 55) * 4),
                        "v"(c0), "v"(c1), "v"(stv) : "memory");
        meta = 0x40u | b3;
        done = true;
      }
    }
    if (!done) {
    *(LAS u32x4*)(scr + lane * 32) = xv0;
    *(LAS u32x4*)(scr + lane * 32 + 16) = xv1;
    if (lane < 16) *(LAS unsigned*)(scr + NIB + lane * 4) = 0u;
    const unsigned cB = (128u - 8u * base) * 0x01010101u, cB7 = (72u - 8u * base) * 0x01010101u;
    const unsigned cLo = (base >= 6u ? 176u - 8u * base : 128u) * 0x01010101u;
    const unsigned cHi = (base <= 6u ? 56u - 8u * base : 0u) * 0x01010101u;
    unsigned C[8], M = 0, far = 0;
#pragma unroll
    for (int i = 0; i < 8; ++i) {
      const unsigned u = D[i] & 0x78787878u;
      const unsigned g1 = u + cB, g2 = u + cB7;
      const unsigned esc = (~g1 | g2) & 0x80808080u;
      far |= ~(u + cLo) | (u + cHi);
      C[i] = (D[i] & 0x87878787u) | (g1 & 0x38383838u) | ((esc - (esc >> 3)) >> 1);
      M |= esc >> (7 - i);
    }
#pragma unroll
    for (int i = 0; i < 7; ++i) C[i] |= shift_pos(C[7], CPOS[i], 6) & 0x40404040u;
    const unsigned cnt = __builtin_popcount(M);
    const unsigned inc = wave_scan(cnt);
    const unsigned total = __builtin_amdgcn_readlane(inc, 63);
    const bool use3 = total <= 85u && !__any((far & 0x80808080u) != 0u);
    if ((use3 || total <= 64u) && !__any(cnt > 8u)) {
      const unsigned wd = use3 ? 3u : 4u;
      unsigned S = 0, Mm = M;
      unsigned by[8];
#pragma unroll
      for (int k = 0; k < 8; ++k) {
        const unsigned bt = (unsigned)__builtin_ctz(Mm | 0x80000000u);
        by[k] = *(LAS unsigned char*)(scr + lane * 32 + ((bt & 7) << 2) + (bt >> 3));
        Mm &= Mm - 1;
      }
#pragma unroll
      for (int k = 0; k < 8; ++k) {
        const unsigned e = (by[k] >> 3) & 15u;
        const int d = (int)e - (int)base;
        const unsigned code = use3 ? (unsigned)(d < 0 ? -1 - d : d - 1) : e;
        S |= (k < (int)cnt ? code : 0u) << (wd * k);
      }
      const unsigned bp = (inc - cnt) * wd, sh = bp & 31u;
      if (cnt) {
        LAS unsigned* sp = (LAS unsigned*)(scr + NIB) + (bp >> 5);
        __hip_atomic_fetch_or(sp, S << sh, __ATOMIC_RELAXED, __HIP_MEMORY_SCOPE_WORKGROUP);
        __hip_atomic_fetch_or(sp + 1, (unsigned)(((unsigned long long)S << sh) >> 32), __ATOMIC_RELAXED, __HIP_MEMORY_SCOPE_WORKGROUP);
      }
      const unsigned stv = *(volatile LAS unsigned*)(scr + NIB + (lane < 8 ? lane : 7) * 4);
      unsigned char* sec = myws + EXPC_OFF + (size_t)q * SHARD_BYTES + (size_t)b * BLK_CMP + w * SEC_CMP;
      typedef unsigned u32x3 __attribute__((ext_vector_type(3)));
      const u32x4 c0 = {C[0], C[1], C[2], C[3]};
      const u32x3 c1 = {C[4], C[5], C[6]};
      asm volatile("global_store_dwordx4 %0, %3, off sc0 sc1\n\tglobal_store_dwordx3 %1, %4, off sc0 sc1\n\t"
                   "global_store_dword %2, %5, off sc0 sc1\n\ts_nop 1"
                   :: "v"(sec + lane * 16), "v"(sec + 1024 + lane * 12), "v"(sec + 1792 + (lane < 8 ? lane : 7) * 4),
                      "v"(c0), "v"(c1), "v"(stv) : "memory");
      meta = (use3 ? 0xC0u : 0x80u) | base;
    } else {
      st16x2_sys(myws + EXPR_OFF + (size_t)q * SHARD_BYTES + (size_t)(b * 4 + w) * SEC_RAW + lane * 16, xv0, xv1);
    }
    }
  }
  TS(1);
  WAIT_VM(0);
  if (lane == 0) *(volatile LAS unsigned*)(misc + w) = tag | meta;
  TS(2);
  if (w == 3) {
    u32x4 fv;
    do {
      asm volatile("ds_read_b128 %0, %1\n\ts_waitcnt lgkmcnt(0)" : "=v"(fv) : "v"(lds0 + LDS_MISC) : "memory");
    } while (fv.x == 0u || fv.y == 0u || fv.z == 0u || fv.w == 0u);
    if (lane >= 1 && lane < NR)
      asm volatile("global_store_dwordx4 %0, %1, off sc0 sc1\n\ts_nop 1"
                   :: "v"((unsigned*)(peerws + FLAG_OFF) + ((q * NR + rank) * 8 + grp) * 128 + idx * 4), "v"(fv) : "memory");
  } else if (w == 0) {
    unsigned bad = 0;
    u32x4 f = {0u, 0u, 0u, 0u};
    if (grp != 0) {
      const unsigned* fptr = (const unsigned*)(myws + FLAG_OFF) + ((q * NR + src) * 8 + rb) * 128 + (lane & 31) * 4;
      const unsigned long long t0 = __builtin_amdgcn_s_memrealtime();
      for (unsigned it = 1;; ++it) {
        asm volatile("global_load_dwordx4 %0, %1, off sc0 sc1\n\ts_waitcnt vmcnt(0)" : "=v"(f) : "v"(fptr) : "memory");
        if (__all(tags_ok(f, tag))) break;
        if (it > 64u) __builtin_amdgcn_s_sleep(8);
        if ((it & 63u) == 0 && (__hip_atomic_load(tmo, __ATOMIC_RELAXED, __HIP_MEMORY_SCOPE_AGENT) == seq ||
                                __builtin_amdgcn_s_memrealtime() - t0 > DEADLINE_TICKS)) {
          if (lane == 0) __hip_atomic_store(tmo, seq, __ATOMIC_RELAXED, __HIP_MEMORY_SCOPE_AGENT);
          bad = 1;
          f = (u32x4){0u, 0u, 0u, 0u};
          break;
        }
      }
    }
    if (lane < 32) *(LAS u32x4*)(misc + 64 + lane * 4) = f;
    if (lane == 0) misc[4] = bad;
    WAIT_LGKM();
  }
  __builtin_amdgcn_s_barrier();
  if (w == 3 && lane == 0) __hip_atomic_store(seqp, seq, __ATOMIC_RELAXED, __HIP_MEMORY_SCOPE_AGENT);
  const unsigned flags = misc[64 + (lane & 31) * 4 + w];
  TS(3);

  const unsigned o1 = grp == 0 ? rowoff : (unsigned)lane * 16;
  const unsigned o2 = grp == 0 ? rowoff + 64 : 1024u + lane * 16;
  const unsigned char* raw_base = grp == 0 ? p.x + (size_t)(rb * BM) * KD
                                           : p.ws[src] + EXPR_OFF + (size_t)q * SHARD_BYTES + (size_t)(rb * 4 + w) * SEC_RAW;
  const unsigned raw_step = grp == 0 ? BK : 8 * 8192;
  const unsigned char* cmp_base = p.ws[src] + EXPC_OFF + (size_t)q * SHARD_BYTES + (size_t)rb * BLK_CMP + w * SEC_CMP;
  const unsigned char* c6_base = p.ws[src] + EXP6_OFF + (size_t)q * SHARD_BYTES + (size_t)rb * BLK_C6 + w * 1024;
  const unsigned o2c = 1024u + lane * 12, o3c = 1792u + (lane < 8 ? lane : 7) * 4;
  const unsigned o2s = 4096u - 512u * w + (lane & ~1) * 8, o3s = 6144u - 800u * w + (lane < 56 ? lane : 55) * 4;
#define LOAD_A(LO, HI, ST, ME, POS) do {                                                                         \
    const int ks_ = (POS);                                                                                       \
    ME = __builtin_amdgcn_readlane(flags, ks_) & 0xFFu;                                                          \
    const bool c_ = (ME & 0x80u) != 0, s_ = (ME & 0xC0u) == 0x40u;                                               \
    const unsigned char* base_ = (s_ ? c6_base : (c_ ? cmp_base : raw_base)) +                                   \
                                 (unsigned)ks_ * (s_ ? 8u * BLK_C6 : (c_ ? 8u * BLK_CMP : raw_step));            \
    const unsigned oa_ = s_ ? o2s : (c_ ? o2c : o2), ob_ = s_ ? o3s : (c_ ? o3c : o1);                           \
    asm volatile("global_load_dwordx4 %0, %3, %6\n\tglobal_load_dwordx4 %1, %4, %6\n\tglobal_load_dword %2, %5, %6" \
                 : "=&v"(LO), "=&v"(HI), "=&v"(ST) : "v"(o1), "v"(oa_), "v"(ob_), "s"(base_) : "memory");        \
  } while (0)
  u32x4 lo0, hi0, lo1, hi1;
  unsigned st0, st1, me0, me1;
  LOAD_A(lo0, hi0, st0, me0, 0);
  LOAD_A(lo1, hi1, st1, me1, 1);
  unsigned awr0, awr1;
  {
    const int r = 16 * w + (lane >> 2), fa = (r >> 1) & 7;
    awr0 = r * 128 + (((lane & 3) ^ fa) << 4);
    awr1 = r * 128 + ((((lane & 3) + 4) ^ fa) << 4);
  }
#define STEP(LO, HI, ST, ME, I, U) do {                                                                          \
    const int i = (I);                                                                                           \
    if (i == 16) TS(44);                                                                                         \
    if (i < NSTEP - 1) WAIT_VM(3); else WAIT_VM(0);                                                              \
    asm volatile("" : "+v"(LO), "+v"(HI), "+v"(ST));                                                             \
    TS(4 + i);                                                                                                   \
    unsigned char* sa = smem + LDS_A + (U) * A_BYTES;                                                            \
    if ((ME & 0xC0u) == 0x40u) {                                                                                 \
      const unsigned b3 = (ME & 15u) * 0x01010101u;                                                              \
                          \
      const unsigned LL = (((b3 + 0x0D0C0B0Au) & 0x0F0F0F0Fu) ^ (b3 + 0x03030303u)) << 3;                        \
      const unsigned LH = (((b3 + 0x14130F0Eu) & 0x0F0F0F0Fu) ^ (b3 + 0x03030303u)) << 3;                        \
      unsigned C[8] = {LO.x, LO.y, LO.z, LO.w, (lane & 1) ? HI.z : HI.x, (lane & 1) ? HI.w : HI.y, 0u, 0u};      \
      _Pragma("unroll") for (int r = 0; r < 2; ++r)                                                             \
        C[6 + r] = ((C[3 * r] >> 5) & 0x03030303u) | ((C[3 * r + 1] >> 3) & 0x0C0C0C0Cu) |                       \
                   ((C[3 * r + 2] << 1) & 0x80808080u) | ((C[3 * r + 2] >> 1) & 0x10101010u);                    \
      unsigned sel[8], run[9];                                                                                   \
      run[0] = 0;                                                                                                \
      _Pragma("unroll") for (int k = 0; k < 8; ++k) {                                                            \
        C[k] &= 0x9F9F9F9Fu;                                                                                     \
        const unsigned f = ((C[k] + 0x08080808u) >> 5) & 0x01010101u;                                            \
        sel[k] = lds_u32(__builtin_amdgcn_udot4(f, 0x20100804u, lds0 + LDS_MISC + 832, false));                  \
        run[k + 1] = __builtin_amdgcn_udot4(f, 0x03030303u, run[k], false);                          \
      }                                                                                                          \
      const unsigned bp = wave_scan(run[8]) - run[8];                                                            \
      const unsigned w0 = __builtin_amdgcn_ds_bpermute((int)((bp >> 5) << 2), (int)ST);                          \
      const unsigned w1 = __builtin_amdgcn_ds_bpermute((int)(((bp >> 5) + 1) << 2), (int)ST);                    \
      const unsigned w2 = __builtin_amdgcn_ds_bpermute((int)(((bp >> 5) + 2) << 2), (int)ST);                    \
      const unsigned long long W = ((unsigned long long)__builtin_amdgcn_alignbit(w2, w1, bp & 31u) << 32) |     \
                                   __builtin_amdgcn_alignbit(w1, w0, bp & 31u);                                  \
      _Pragma("unroll") for (int k = 0; k < 8; ++k) {                                                            \
        const unsigned v = (unsigned)(W >> run[k]);                                                              \
        const unsigned y = (((v & 0x1C7u) * 0x401u) & 0x00070007u) |                  \
                           (((v & 0xE38u) * 0x8020u) & 0x07000700u);                  \
        const unsigned z = __builtin_amdgcn_perm(y, 0x0C0C0C0Cu, sel[k]);                                        \
        C[k] = (C[k] + (b3 << 3)) ^ __builtin_amdgcn_perm(LH, LL, z);                                            \
      }                                                                                                          \
      *(u32x4*)(sa + awr0) = (u32x4){C[0], C[1], C[2], C[3]};                                                    \
      *(u32x4*)(sa + awr1) = (u32x4){C[4], C[5], C[6], C[7]};                                                    \
    } else if (ME & 0x80u) {                                                                                     \
      const unsigned base = ME & 15u;                                                                            \
      unsigned C[8] = {LO.x, LO.y, LO.z, LO.w, HI.x, HI.y, HI.z, 0u};                                            \
      _Pragma("unroll") for (int k = 0; k < 7; ++k) C[7] |= shift_pos(C[k] & 0x40404040u, 6, CPOS[k]);           \
      unsigned M = 0, Dd[8];                                                                                     \
      _Pragma("unroll") for (int k = 0; k < 8; ++k) {                                                            \
        const unsigned t = C[k] & 0xBFBFBFBFu;                                                                   \
        M |= shift_pos((t + 0x08080808u) & 0x40404040u, 6, k);                                                   \
        Dd[k] = t + base * 0x08080808u;                                                                          \
      }                                                                                                          \
      *(u32x4*)(sa + awr0) = (u32x4){Dd[0], Dd[1], Dd[2], Dd[3]};                                                \
      *(u32x4*)(sa + awr1) = (u32x4){Dd[4], Dd[5], Dd[6], Dd[7]};                                                \
      const unsigned cnt = __builtin_popcount(M);                                                                \
      const bool use3 = (ME & 0x40u) != 0;                                                                       \
      const unsigned wd = use3 ? 3u : 4u;                                                                        \
      const unsigned bp = (wave_scan(cnt) - cnt) * wd;                                                           \
      const unsigned w0 = __builtin_amdgcn_ds_bpermute((int)((bp >> 5) << 2), (int)ST);                          \
      const unsigned w1 = __builtin_amdgcn_ds_bpermute((int)(((bp >> 5) + 1) << 2), (int)ST);                    \
      unsigned S = __builtin_amdgcn_alignbit(w1, w0, bp & 31u);                                                  \
                          \
      const unsigned abase = (unsigned)(uintptr_t)(LAS unsigned char*)sa + awr0;                                 \
      if (use3) {                                                                                                \
        while (__any(M != 0u)) {                                                                                 \
          const unsigned bt = (unsigned)__builtin_ctz(M | 0x80000000u);                                          \
          const unsigned c = S & 7u;                                                                             \
          unsigned dl = c < 6u ? 0u - 8u - c : c - 6u;                                                           \
          dl = M ? dl : 0u;                                                                                      \
          M &= M - 1;                                                                                            \
          S >>= 3;                                                                                               \
          asm volatile("ds_add_u32 %0, %1" :: "v"(abase ^ (((bt << 4) & 64u) | ((bt << 2) & 12u))),              \
                       "v"(dl << ((bt & 24u) | 3u)) : "memory");                                                 \
        }                                                                                                        \
      } else {                                                                                                   \
        while (__any(M != 0u)) {                                                                                 \
          const unsigned bt = (unsigned)__builtin_ctz(M | 0x80000000u);                                          \
          const unsigned dl = M ? (S & 15u) - base - 7u : 0u;                                                    \
          M &= M - 1;                                                                                            \
          S >>= 4;                                                                                               \
          asm volatile("ds_add_u32 %0, %1" :: "v"(abase ^ (((bt << 4) & 64u) | ((bt << 2) & 12u))),              \
                       "v"(dl << ((bt & 24u) | 3u)) : "memory");                                                 \
        }                                                                                                        \
      }                                                                                                          \
    } else {                                                                                                     \
      *(u32x4*)(sa + awr0) = LO;                                                                                 \
      *(u32x4*)(sa + awr1) = HI;                                                                                 \
    }                                                                                                            \
    WAIT_LGKM();                                                               \
    if (i + 2 < NSTEP) LOAD_A(LO, HI, ST, ME, i + 2);                                                            \
    if (i == 16) TS(45);                                                                                         \
    __builtin_amdgcn_s_barrier();                                                                         \
    if (i == 16) TS(46);                                                                                         \
  } while (0)
#pragma unroll 1
  for (int ii = 0; ii < NSTEP; ii += 2) {
    STEP(lo0, hi0, st0, me0, ii, 0);
    STEP(lo1, hi1, st1, me1, ii + 1, 1);
  }
  TS_DUMP();
}

}

extern "C" void kernel_launch(int rank, int nranks, ncclComm_t comm,
                              hipStream_t stream,
                              void* const* d_in, const int* in_sizes, int n_in,
                              void* d_out, int out_size,
                              const int* in_dtypes, int out_dtype,
                              void* const* d_out_all,
                              void* const* d_ws_all, size_t ws_bytes) {
  if (nranks != NR || ws_bytes < WS_NEED || n_in != 4) { fprintf(stderr, "unexpected configuration\n"); return; }
  HIP_CHECK(hipFuncSetAttribute((const void*)ag_gemm_kernel, hipFuncAttributeMaxDynamicSharedMemorySize, LDS_BYTES));
  Params p;
  p.x = (const unsigned char*)d_in[0];
  p.w = (const unsigned char*)d_in[1];
  p.sx = (const float*)d_in[2];
  p.sw = (const float*)d_in[3];
  p.out = (float*)d_out;
  for (int i = 0; i < NR; ++i) p.ws[i] = (unsigned char*)d_ws_all[i];
  p.myws = (unsigned char*)d_ws_all[rank];
  p.rank = rank;
  ag_gemm_kernel<<<NWG, NTHR, LDS_BYTES, stream>>>(p);
  HIP_CHECK(hipGetLastError());
}
```
